```python
import numpy as np

kernel_name = "gpu_ag_gemm_m8192_k8192_n8192_f32_gelu"


ABSMAX_RTOL = 0.001

COMPUTE_DTYPE = "float32"
EPILOGUE = "gelu"
IN_QUANT = None
OUT_QUANT = None
IN_DTYPES = {"x": "float32", "w_mat": "float32"}
WORLD_SIZE = 8

def setup_inputs():

    x = np.random.randn(8192, 8192).astype(np.float32)
    w_mat = (np.random.randn(8192, 8192) / np.sqrt(8192)).astype(np.float32)
    return {"x": x, "w_mat": w_mat}

def _epilogue(y):
    c = 0.7978845608028654
    return (0.5 * y * (1.0 + np.tanh(c * (y + 0.044715 * y ** 3)))).astype(np.float32)

def reference(x, w_mat):
    y = (x @ w_mat).astype(np.float32)
    return _epilogue(y)
```

```cpp
#include <hip/hip_runtime.h>
#include <rccl/rccl.h>
#include <cstdio>
#include <cstdint>

#define HIP_CHECK(expr) do { hipError_t _e = (expr); if (_e != hipSuccess) { \
    fprintf(stderr, "HIP error %s at %s:%d\n", hipGetErrorString(_e), __FILE__, __LINE__); \
    return; } } while (0)

typedef unsigned u32;
typedef unsigned long long u64;
typedef _Float16 h8 __attribute__((ext_vector_type(8)));
typedef float f4 __attribute__((ext_vector_type(4)));
typedef unsigned u4 __attribute__((ext_vector_type(4)));
#define GAS __attribute__((address_space(1)))
#define LAS __attribute__((address_space(3)))

namespace {

constexpr int NR = 8;
constexpr int KDIM = 8192, NLOC = 1024;
constexpr int NKT = KDIM / 64;
constexpr size_t CTR_OFF = 0;
constexpr size_t TMO_OFF = 32768;
constexpr size_t FLG_OFF = 65536;
constexpr size_t DON_OFF = 98304;
constexpr size_t DBG_OFF = 131072;
constexpr size_t DATA_OFF = 1u << 20;
constexpr size_t KT_B = 131072;
constexpr size_t STREAM_B = (size_t)NKT * KT_B;
constexpr int NSTREAM = 10;
constexpr size_t PAR_B = NSTREAM * STREAM_B;
constexpr size_t WS_NEED = DATA_OFF + 2 * PAR_B;
constexpr int LDS_B = 131072;
constexpr u64 DEADLINE = 50000000ull;

struct Params {
  const float* x; const float* w;
  char* ws[NR]; float* out[NR];
  int rank; int pad;
};

#define FENCE() asm volatile("" ::: "memory")
__device__ __forceinline__ void raw_barrier() { FENCE(); __builtin_amdgcn_s_barrier(); FENCE(); }
template <int N> __device__ __forceinline__ void wait_vm() { asm volatile("s_waitcnt vmcnt(%0)" :: "n"(N) : "memory"); }

__device__ __forceinline__ u32 ld_sys(const GAS u32* p) { return __hip_atomic_load(p, __ATOMIC_RELAXED, __HIP_MEMORY_SCOPE_SYSTEM); }
__device__ __forceinline__ u32 ld_agent(const GAS u32* p) { return __hip_atomic_load(p, __ATOMIC_RELAXED, __HIP_MEMORY_SCOPE_AGENT); }
__device__ __forceinline__ void st_sys(GAS u32* p, u32 v) { __hip_atomic_store(p, v, __ATOMIC_RELAXED, __HIP_MEMORY_SCOPE_SYSTEM); }
__device__ __forceinline__ void st_agent(GAS u32* p, u32 v) { __hip_atomic_store(p, v, __ATOMIC_RELAXED, __HIP_MEMORY_SCOPE_AGENT); }

template <int OFF> __device__ __forceinline__ void st16_wt(const char* base, u32 voff, u4 v) {
  asm volatile("global_store_dwordx4 %0, %1, %2 offset:%3 sc0 sc1\n\ts_nop 1" :: "v"(voff), "v"(v), "s"(base), "n"(OFF) : "memory");
}
template <int OFF> __device__ __forceinline__ void ld16(f4& d, const char* base, u32 voff) {
  asm volatile("global_load_dwordx4 %0, %1, %2 offset:%3" : "=v"(d) : "v"(voff), "s"(base), "n"(OFF) : "memory");
}
__device__ __forceinline__ u4 cvt8(const f4& a, const f4& b) {
  h8 r; r[0] = (_Float16)a[0]; r[1] = (_Float16)a[1]; r[2] = (_Float16)a[2]; r[3] = (_Float16)a[3];
  r[4] = (_Float16)b[0]; r[5] = (_Float16)b[1]; r[6] = (_Float16)b[2]; r[7] = (_Float16)b[3];
  return __builtin_bit_cast(u4, r);
}
__device__ __forceinline__ u4 cvt8s(float a0, float a1, float a2, float a3, float a4, float a5, float a6, float a7) {
  h8 r; r[0] = (_Float16)a0; r[1] = (_Float16)a1; r[2] = (_Float16)a2; r[3] = (_Float16)a3;
  r[4] = (_Float16)a4; r[5] = (_Float16)a5; r[6] = (_Float16)a6; r[7] = (_Float16)a7;
  return __builtin_bit_cast(u4, r);
}

#define STAMP(i) do { } while (0)

template <int ND>
__device__ __forceinline__ void x_stream(const float* x, char* const (&dst)[ND], char* const (&flg)[ND],
                                         int rb0, int nph, int sub, int nsub, u32 cbase) {
  const int wid = __builtin_amdgcn_readfirstlane(threadIdx.x >> 6), lane = threadIdx.x & 63;
  const int fr = lane >> 2, fq = (lane & 3) ^ ((-(lane >> 4)) & 3);
  const u32 lsrc = (u32)(fr * KDIM + 4 * fq) * 4u, ldst = (u32)lane * 16u;
  const int per = 16 / nsub, n = nph * per;
  u64 myflag = (u64)flg[0];
#pragma unroll
  for (int d = 1; d < ND; ++d) { myflag = lane == d ? (u64)flg[d] : myflag; asm volatile("" : "+v"(myflag)); }
  auto src_of = [&](int i) { const int rb = rb0 + 16 * (i / per), it = sub + nsub * (i % per);
    return (const char*)x + ((size_t)rb * 16 * KDIM + (size_t)(8 * it + wid) * 64) * 4; };
  auto off_of = [&](int i) { const int rb = rb0 + 16 * (i / per), it = sub + nsub * (i % per);
    return (size_t)(8 * it + wid) * KT_B + (size_t)rb * 2048; };
  auto publish = [&](int i) { const int rb = rb0 + 16 * (i / per), it = sub + nsub * (i % per);
    int v = 8 * (it + nsub); v = v > NKT ? NKT : v;
    if (threadIdx.x < ND) st_sys((GAS u32*)(myflag + (u64)((rb * 4 + sub) * 4)), cbase + (u32)v); };
  f4 ra[4], rb_[4];
#define XLOAD(R, i) do { const char* s_ = src_of(i); ld16<0>(R[0], s_, lsrc); ld16<64>(R[1], s_, lsrc); ld16<128>(R[2], s_, lsrc); ld16<192>(R[3], s_, lsrc); } while (0)
#define XSTORE(R, i) do { const u4 c0 = cvt8(R[0], R[1]), c1 = cvt8(R[2], R[3]); const size_t o_ = off_of(i); \
    _Pragma("unroll") for (int d = 0; d < ND; ++d) { st16_wt<0>(dst[d] + o_, ldst, c0); st16_wt<1024>(dst[d] + o_, ldst, c1); } } while (0)
#define XWAIT(R, N) asm volatile("s_waitcnt vmcnt(%4)" : "+v"(R[0]), "+v"(R[1]), "+v"(R[2]), "+v"(R[3]) : "n"(N) : "memory")
  XLOAD(ra, 0);
  XLOAD(rb_, 1); XWAIT(ra, 4); XSTORE(ra, 0);
  for (int i = 1; i < n - 1; i += 2) {
    XLOAD(ra, i + 1); XWAIT(rb_, 4 + 2 * ND); XSTORE(rb_, i);
    wait_vm<4 + 2 * ND>(); raw_barrier(); publish(i - 1);
    if (i + 2 < n) {
      XLOAD(rb_, i + 2); XWAIT(ra, 4 + 2 * ND); XSTORE(ra, i + 1);
      wait_vm<4 + 2 * ND>(); raw_barrier(); publish(i);
    }
  }
  XWAIT(rb_, 2 * ND); XSTORE(rb_, n - 1);
  wait_vm<2 * ND>(); raw_barrier(); publish(n - 2);
  wait_vm<0>(); raw_barrier(); publish(n - 1);
#undef XLOAD
#undef XSTORE
#undef XWAIT
}

__device__ __forceinline__ void w_stream(const float* w, char* dst, char* flg, int gsrc, int gdst,
                                         int sub, int nsub, u32 cbase) {
  const int wid = __builtin_amdgcn_readfirstlane(threadIdx.x >> 6), lane = threadIdx.x & 63;
  const int fr = lane & 15, fq = lane >> 4;
  const u32 l0 = (u32)(4 * fq * NLOC + 4 * fr) * 4u, ldst = (u32)lane * 16u;
  const u32 l1 = l0 + NLOC * 4, l2 = l0 + 2 * NLOC * 4, l3 = l0 + 3 * NLOC * 4;
  const int n = 32 / nsub;
  auto src_of = [&](int i) { const int kb = 8 * (sub + nsub * i) + wid;
    return (const char*)w + ((size_t)kb * 32 * NLOC + (size_t)gsrc * 64) * 4; };
  auto dst_of = [&](int i) { const int kb = 8 * (sub + nsub * i) + wid;
    return dst + (size_t)(kb >> 1) * KT_B + (size_t)gdst * 8192 + (kb & 1) * 1024; };
  auto publish = [&](int i) { int v = 4 * (sub + nsub * i + nsub); v = v > NKT ? NKT : v;
    if (threadIdx.x == 0) st_sys((GAS u32*)(flg + (gdst * 4 + sub) * 4), cbase + (u32)v); };
  f4 ra[8], rb_[8];
#define WLOAD(R, i) do { const char* s_ = src_of(i); const char* t_ = s_ + 16 * NLOC * 4; \
    ld16<0>(R[0], s_, l0); ld16<0>(R[1], s_, l1); ld16<0>(R[2], s_, l2); ld16<0>(R[3], s_, l3); \
    ld16<0>(R[4], t_, l0); ld16<0>(R[5], t_, l1); ld16<0>(R[6], t_, l2); ld16<0>(R[7], t_, l3); } while (0)
#define WCV(R, j) cvt8s(R[0][j], R[1][j], R[2][j], R[3][j], R[4][j], R[5][j], R[6][j], R[7][j])
#define WSTORE(R, i) do { const char* d_ = dst_of(i); const char* e_ = d_ + 4096; \
    st16_wt<0>(d_, ldst, WCV(R, 0)); st16_wt<2048>(d_, ldst, WCV(R, 1)); st16_wt<0>(e_, ldst, WCV(R, 2)); st16_wt<2048>(e_, ldst, WCV(R, 3)); } while (0)
#define WWAIT(R, N) asm volatile("s_waitcnt vmcnt(%8)" : "+v"(R[0]), "+v"(R[1]), "+v"(R[2]), "+v"(R[3]), "+v"(R[4]), "+v"(R[5]), "+v"(R[6]), "+v"(R[7]) : "n"(N) : "memory")
  WLOAD(ra, 0);
  WLOAD(rb_, 1); WWAIT(ra, 8); WSTORE(ra, 0);
  for (int i = 1; i < n - 1; i += 2) {
    WLOAD(ra, i + 1); WWAIT(rb_, 12); WSTORE(rb_, i);
    wait_vm<12>(); raw_barrier(); publish(i - 1);
    if (i + 2 < n) {
      WLOAD(rb_, i + 2); WWAIT(ra, 12); WSTORE(ra, i + 1);
      wait_vm<12>(); raw_barrier(); publish(i);
    }
  }
  WWAIT(rb_, 4); WSTORE(rb_, n - 1);
  wait_vm<4>(); raw_barrier(); publish(n - 2);
  wait_vm<0>(); raw_barrier(); publish(n - 1);
#undef WLOAD
#undef WCV
#undef WSTORE
#undef WWAIT
}

struct Task {
  const char* a;
  const char* b;
  const char* fa;
  const char* fb;
  int nsa, nsb;
  float* out;
  char* done;
};

__device__ __forceinline__ float gelu(float y) {
  const float t = y * (y * y * 0.044715f + 1.0f) * -2.302208198f;
  return y * __builtin_amdgcn_rcpf(1.0f + __builtin_amdgcn_exp2f(t));
}

#define GLDS_BITS " sc0 sc1"

__device__ __forceinline__ void glds4(const char* g, u32 voff, u32 ldsdst) {
  u32 keep;
  asm volatile("s_mov_b32 %0, m0\n\ts_mov_b32 m0, %3\n\ts_nop 0\n\t"
               "global_load_lds_dwordx4 %1, %2" GLDS_BITS "\n\t"
               "global_load_lds_dwordx4 %1, %2 offset:1024" GLDS_BITS "\n\t"
               "global_load_lds_dwordx4 %1, %2 offset:2048" GLDS_BITS "\n\t"
               "global_load_lds_dwordx4 %1, %2 offset:3072" GLDS_BITS "\n\t"
               "s_mov_b32 m0, %0"
               : "=&s"(keep) : "v"(voff), "s"(g), "s"(ldsdst) : "memory");
}

__device__ __forceinline__ int poll_ready(const GAS u32* fp, GAS u32* tmo, u32 cbase, u32 call, int need) {
  const u64 t0 = __builtin_amdgcn_s_memrealtime();
  int r;
  for (;;) {
    int d = (int)(ld_sys(fp) - cbase);
    d = d < 0 ? 0 : (d > NKT ? NKT : d);
#pragma unroll
    for (int o = 32; o >= 1; o >>= 1) { const int e = __shfl_xor(d, o); d = e < d ? e : d; }
    r = __builtin_amdgcn_readfirstlane(d);
    if (r >= need) break;
    if (__builtin_amdgcn_readfirstlane(ld_agent(tmo)) == call || __builtin_amdgcn_s_memrealtime() - t0 > DEADLINE) {
      st_agent(tmo, call); r = NKT; break;
    }
    __builtin_amdgcn_s_sleep(8);
  }
  return r;
}

__device__ __forceinline__ void gemm_task(LAS char* lds, const Task& t, GAS u32* tmo, u32 cbase, u32 call) {
  const int wid = __builtin_amdgcn_readfirstlane(threadIdx.x >> 6), lane = threadIdx.x & 63;
  const int kh = wid >> 2, wm = (wid >> 1) & 1, wn = wid & 1, fr = lane & 15, fq = lane >> 4;
  const u32 lds0 = (u32)(uintptr_t)lds;
  const u32 voff = (u32)lane * 16u;
  const char* g = (wid < 4 ? t.a : t.b) + (wid & 3) * 4096;
  const u32 sdst = lds0 + (u32)wid * 4096u;
  const GAS u32* fp = (const GAS u32*)(lane < 32 ? t.fa + ((lane >> 2) * 4 + ((lane & 3) & (t.nsa - 1))) * 4
                                                 : t.fb + (((lane >> 4) & 1) * 4 + ((lane & 3) & (t.nsb - 1))) * 4);
  const LAS char* pa = lds + wm * 8192 + kh * 1024 + (fr * 4 + (fq ^ ((-(fr >> 2)) & 3))) * 16;
  const LAS char* pb = lds + 16384 + wn * 8192 + kh * 1024 + lane * 16;
  f4 acc[4][4];
#pragma unroll
  for (int m = 0; m < 4; ++m)
#pragma unroll
    for (int j = 0; j < 4; ++j) acc[m][j] = (f4){0.f, 0.f, 0.f, 0.f};
  h8 A0[4], B0[4], A1[4], B1[4];
  int ready = 0;
#define RD(A, B, buf) do { _Pragma("unroll") for (int m = 0; m < 4; ++m) A[m] = *(const LAS h8*)(pa + (buf) * 32768 + m * 2048); \
    _Pragma("unroll") for (int j = 0; j < 4; ++j) B[j] = *(const LAS h8*)(pb + (buf) * 32768 + j * 2048); } while (0)
#define MM(A, B) do { _Pragma("unroll") for (int m = 0; m < 4; ++m) _Pragma("unroll") for (int j = 0; j < 4; ++j) \
    acc[m][j] = __builtin_amdgcn_mfma_f32_16x16x32_f16(A[m], B[j], acc[m][j], 0, 0, 0); } while (0)
  if (wid == 0) ready = poll_ready(fp, tmo, cbase, call, 4);
  raw_barrier();
  glds4(g, voff, sdst); glds4(g + KT_B, voff, sdst + 32768); glds4(g + 2 * KT_B, voff, sdst + 65536); glds4(g + 3 * KT_B, voff, sdst + 98304);
  wait_vm<12>(); raw_barrier();
  RD(A0, B0, 0);
#define STEP(s, buf, AC, BC, AN, BN) do { \
    const int nx = (s) + 4 < NKT ? (s) + 4 : NKT - 1; \
    if (wid == 0 && nx >= ready) ready = poll_ready(fp, tmo, cbase, call, nx + 1); \
    FENCE(); __builtin_amdgcn_s_waitcnt(0x0078);   wait_vm<8>(); raw_barrier(); \
    glds4(g + (size_t)nx * KT_B, voff, sdst + (buf) * 32768); \
    RD(AN, BN, ((buf) + 1) & 3); FENCE(); __builtin_amdgcn_sched_barrier(0); \
    MM(AC, BC); __builtin_amdgcn_sched_barrier(0); } while (0)
  for (int s = 0; s < NKT; s += 4) {
    STEP(s, 0, A0, B0, A1, B1);
    STEP(s + 1, 1, A1, B1, A0, B0);
    STEP(s + 2, 2, A0, B0, A1, B1);
    STEP(s + 3, 3, A1, B1, A0, B0);
  }
#undef STEP
#undef RD
#undef MM
  const u32 bad = ld_agent(tmo);
  FENCE(); __builtin_amdgcn_s_waitcnt(0x0070); wait_vm<0>(); raw_barrier();
  LAS f4* ex = (LAS f4*)lds + wid * 512 + lane;
  LAS f4* ey = (LAS f4*)lds + (wid ^ 4) * 512 + lane;
  f4 v[2][4];
  if (kh == 0) {
#pragma unroll
    for (int m = 0; m < 2; ++m)
#pragma unroll
      for (int j = 0; j < 4; ++j) { ex[(m * 4 + j) * 64] = acc[2 + m][j]; v[m][j] = acc[m][j]; }
  } else {
#pragma unroll
    for (int m = 0; m < 2; ++m)
#pragma unroll
      for (int j = 0; j < 4; ++j) { ex[(m * 4 + j) * 64] = acc[m][j]; v[m][j] = acc[2 + m][j]; }
  }
  FENCE(); __builtin_amdgcn_s_waitcnt(0xC07F); raw_barrier();
#pragma unroll
  for (int m = 0; m < 2; ++m)
#pragma unroll
    for (int j = 0; j < 4; ++j) v[m][j] += ey[(m * 4 + j) * 64];
  const bool poison = bad == call;
  const char* ob = (const char*)(t.out + (size_t)(wm * 64 + kh * 32) * NLOC + wn * 64);
  const u32 oo = (u32)(4 * fq * NLOC + 4 * fr) * 4u;
#pragma unroll
  for (int m = 0; m < 2; ++m)
#pragma unroll
    for (int i = 0; i < 4; ++i) {
      f4 o = {gelu(v[m][0][i]), gelu(v[m][1][i]), gelu(v[m][2][i]), gelu(v[m][3][i])};
      if (poison) o = (f4){__builtin_nanf(""), __builtin_nanf(""), __builtin_nanf(""), __builtin_nanf("")};
      st16_wt<0>(ob + (size_t)(m * 16 + i) * NLOC * 4, oo, __builtin_bit_cast(u4, o));
    }
  if (t.done) wait_vm<0>();
  FENCE(); __builtin_amdgcn_s_waitcnt(0xC07F); raw_barrier();
  if (t.done && threadIdx.x == 0) st_sys((GAS u32*)t.done, call);
}

__global__ __launch_bounds__(512, 2) void ag_gemm(Params p) {
  extern __shared__ __attribute__((aligned(1024))) char smem[];
  LAS char* lds = (LAS char*)smem;
  const int rank = p.rank & 7, me = rank & 3, og = (rank & 4) ^ 4;
  char* ws = p.ws[rank];
  STAMP(0);
  u32 call;
  {
    GAS u32* ctr = (GAS u32*)(ws + CTR_OFF + (size_t)blockIdx.x * 128);
    call = (u32)__builtin_amdgcn_readfirstlane(ld_agent(ctr)) + 1u;
    asm volatile("" : "+s"(call) :: "memory");
    raw_barrier();
    if (threadIdx.x == 0) st_agent(ctr, call);
  }
  const u32 cbase = call << 8;
  const size_t par = DATA_OFF + (size_t)(call & 1u) * PAR_B;
  GAS u32* tmo = (GAS u32*)(ws + TMO_OFF);
  const int xcd = blockIdx.x & 7, slot = blockIdx.x >> 3;

  if (slot < 2) {
    char* dst[3]; char* flg[3];
#pragma unroll
    for (int a = 0; a < 3; ++a) { char* pw = p.ws[rank ^ (a + 1)]; dst[a] = pw + par + (size_t)rank * STREAM_B; flg[a] = pw + FLG_OFF + rank * 1024; }
    x_stream<3>(p.x, dst, flg, slot * 8 + xcd, 3, 0, 1, cbase);
    STAMP(1);
    if (blockIdx.x == 0) {
      const int wid = __builtin_amdgcn_readfirstlane(threadIdx.x >> 6), lane = threadIdx.x & 63;
      if (wid < 4) {
        const GAS u32* dp = (const GAS u32*)(ws + DON_OFF + (og + wid) * 256) + ((lane & 31) < 30 ? lane : (lane & 32));
        const u64 t0 = __builtin_amdgcn_s_memrealtime();
        for (;;) {
          const bool ok = (int)(ld_sys(dp) - call) >= 0;
          if (__all(ok)) break;
          if (__builtin_amdgcn_readfirstlane(ld_agent(tmo)) == call || __builtin_amdgcn_s_memrealtime() - t0 > DEADLINE) {
            st_agent(tmo, call);
            if (lane == 0) ((GAS float*)p.out[rank])[(size_t)rank * NLOC * NLOC] = __builtin_nanf("");
            break;
          }
          __builtin_amdgcn_s_sleep(4);
        }
      }
      STAMP(2);
    }
    return;
  }
  if (slot < 6) {
    const int o = (slot - 2) * 8 + xcd, q = o >> 3, gq = (o >> 1) & 3, sub = o & 1;
    char* pw = p.ws[og + q];
    w_stream(p.w, pw + par + 9 * STREAM_B, pw + FLG_OFF + 9 * 1024, 4 * q + gq, 4 * me + gq, sub, 2, cbase);
    STAMP(1);
    char* dst[4]; char* flg[4];
#pragma unroll
    for (int a = 0; a < 4; ++a) { char* pv = p.ws[og + ((a + q) & 3)]; dst[a] = pv + par + (size_t)rank * STREAM_B; flg[a] = pv + FLG_OFF + rank * 1024; }
    x_stream<4>(p.x, dst, flg, 48 + (o >> 1), 1, sub, 2, cbase);
    STAMP(2);
    return;
  }
  const int li = slot - 6, gi = li * 8 + xcd;
  if (gi < 64) {
    char* dst[1] = {ws + par + (size_t)rank * STREAM_B}; char* flg[1] = {ws + FLG_OFF + rank * 1024};
    x_stream<1>(p.x, dst, flg, gi, 1, 0, 1, cbase);
  } else if (gi < 128) {
    const int g = (gi - 64) >> 2;
    w_stream(p.w, ws + par + 8 * STREAM_B, ws + FLG_OFF + 8 * 1024, g, g, gi & 3, 4, cbase);
  }
  STAMP(1);
  const int hp = xcd & 1, q = xcd >> 1;
  const int ntask = li < 12 ? 3 : 2;
  for (int ph = 0; ph < ntask; ++ph) {
    int src, panel, c, dn = 0;
    if (ph == 0) {
      if (li < 14) { const int r = li >> 1; c = li & 1; if (r < 4) { src = rank; panel = r; } else { src = rank ^ (r - 3); panel = 0; } }
      else { const int j = li - 14, r = j >> 1; c = 2 + (j & 1); dn = j; if (r < 3) { src = rank; panel = r; } else { src = rank ^ (r - 2); panel = 0; } }
    } else if (li < 12) {
      const int a = li >> 2; c = li & 3; src = rank ^ (a + 1); panel = ph; dn = (ph == 1 ? 12 : 24) + a * 2 + (c & 1);
    } else {
      const int j = li - 12; panel = 3;
      if (j < 8) { src = og + (j >> 1); c = j & 1; }
      else { const int jj = j - 8, s = jj >> 1; src = og + s + (s >= q ? 1 : 0); c = 2 + (jj & 1); dn = 18 + jj; }
    }
    src &= 7;
    const int rb = panel * 16 + hp * 8, grp = 4 * q + 2 * (c & 1), wstream = 8 + (c >> 1);
    Task t;
    t.a = ws + par + (size_t)src * STREAM_B + rb * 2048;
    t.b = ws + par + (size_t)wstream * STREAM_B + grp * 8192;
    t.fa = ws + FLG_OFF + src * 1024 + rb * 16;
    t.fb = ws + FLG_OFF + wstream * 1024 + grp * 16;
    t.nsa = (panel == 3 && src != rank) ? 2 : 1;
    t.nsb = c < 2 ? 4 : 2;
    const size_t row = (size_t)src * NLOC + rb * 16;
    if (c < 2) { t.out = p.out[rank] + row * NLOC + 256 * q + 128 * c; t.done = nullptr; }
    else { t.out = p.out[og + q] + row * NLOC + 256 * me + 128 * (c & 1); t.done = p.ws[og + q] + DON_OFF + rank * 256 + (hp * 32 + dn) * 4; }
    gemm_task(lds, t, tmo, cbase, call);
    STAMP(2 + ph);
  }
}

}

extern "C" void kernel_launch(int rank, int nranks, ncclComm_t comm,
                              hipStream_t stream,
                              void* const* d_in, const int* in_sizes, int n_in,
                              void* d_out, int out_size,
                              const int* in_dtypes, int out_dtype,
                              void* const* d_out_all,
                              void* const* d_ws_all, size_t ws_bytes) {
  if (nranks != NR || n_in != 2 || ws_bytes < WS_NEED || (unsigned)rank >= (unsigned)NR ||
      in_sizes[0] != NLOC * KDIM || in_sizes[1] != KDIM * NLOC || out_size != KDIM * NLOC) {
    fprintf(stderr, "unexpected problem shape\n");
    return;
  }
  static bool attr = false;
  if (!attr) {
    HIP_CHECK(hipFuncSetAttribute((const void*)ag_gemm, hipFuncAttributeMaxDynamicSharedMemorySize, LDS_B));
    attr = true;
  }
  Params p;
  p.x = (const float*)d_in[0]; p.w = (const float*)d_in[1];
  for (int r = 0; r < NR; ++r) { p.ws[r] = (char*)d_ws_all[r]; p.out[r] = (float*)d_out_all[r]; }
  p.rank = rank; p.pad = 0;
  hipLaunchKernelGGL(ag_gemm, dim3(256), dim3(512), LDS_B, stream, p);
  HIP_CHECK(hipGetLastError());
}
```

```cpp
#include <hip/hip_runtime.h>
#include <rccl/rccl.h>
#include <cstdio>
#include <cstdint>

#define HIP_CHECK(expr) do { hipError_t _e = (expr); if (_e != hipSuccess) { \
    fprintf(stderr, "HIP error %s at %s:%d\n", hipGetErrorString(_e), __FILE__, __LINE__); \
    return; } } while (0)

typedef unsigned u32;
typedef unsigned long long u64;
typedef _Float16 h8 __attribute__((ext_vector_type(8)));
typedef float f4 __attribute__((ext_vector_type(4)));
typedef unsigned u4 __attribute__((ext_vector_type(4)));
#define GAS __attribute__((address_space(1)))
#define LAS __attribute__((address_space(3)))

#define NC_SLOTS 4
#define LAG_WY 10
#define LAG_O 2
#define NF 16
#define UB_UNITS 80
#define POLL_SLEEP 8

namespace {

constexpr int NR = 8;
constexpr int KDIM = 8192, NLOC = 1024;
constexpr int NKT = KDIM / 64;
constexpr int NC = 8 * NC_SLOTS, NCS = NC_SLOTS;
constexpr int NGS = 26;
static_assert(NCS + NGS <= 32, "grid");
static_assert(LAG_O <= 2, "the in-group-only iterations publish at once");
#define N_LOCAL 112
constexpr int NLOCAL = N_LOCAL;
constexpr int NST = 60;
constexpr int NSU = 48 * 8 + 12 * 4;
constexpr int UI = 1536;
constexpr int UW = 512, UX = 1024, UO = UX + NSU;
constexpr int UA = UI - UB_UNITS;
constexpr size_t CTR_OFF = 0;
constexpr size_t TMO_OFF = 32768;
constexpr size_t CLM_OFF = 32768 + 256;
constexpr size_t OST_OFF = 36864;
constexpr size_t DON_OFF = 40960;
constexpr size_t FLG_OFF = 65536;
constexpr size_t FLG_B = 8192;
constexpr size_t DBG_OFF = 262144;
constexpr size_t DATA_OFF = 1u << 20;
constexpr size_t KT_B = 131072;
constexpr size_t STREAM_B = (size_t)NKT * KT_B;
constexpr int NSTREAM = 10;
constexpr size_t PAR_B = NSTREAM * STREAM_B;
constexpr size_t STG_OFF = DATA_OFF + 2 * PAR_B;
constexpr size_t WS_NEED = STG_OFF + (size_t)4 * NSU * 8192;
constexpr int LDS_B = 131072;
constexpr u64 DEADLINE = 50000000ull;

struct Params {
  const float* x; const float* w;
  char* ws[NR]; float* out[NR];
  int rank; int pad;
};

#define FENCE() asm volatile("" ::: "memory")
#define SB() __builtin_amdgcn_sched_barrier(0)
__device__ __forceinline__ void raw_barrier() { FENCE(); __builtin_amdgcn_s_barrier(); FENCE(); }
template <int N> __device__ __forceinline__ void wait_vm() { asm volatile("s_waitcnt vmcnt(%0)" :: "n"(N) : "memory"); }
__device__ __forceinline__ void wait_lgkm() { FENCE(); __builtin_amdgcn_s_waitcnt(0xC07F); FENCE(); }
constexpr int waitcnt(int vm, int lgkm) { return (vm & 15) | (7 << 4) | ((lgkm & 15) << 8) | ((vm >> 4) << 14); }

__device__ __forceinline__ u32 ld_sys(const GAS u32* p) { return __hip_atomic_load(p, __ATOMIC_RELAXED, __HIP_MEMORY_SCOPE_SYSTEM); }
__device__ __forceinline__ u32 ld_agent(const GAS u32* p) { return __hip_atomic_load(p, __ATOMIC_RELAXED, __HIP_MEMORY_SCOPE_AGENT); }
__device__ __forceinline__ void st_sys(GAS u32* p, u32 v) { __hip_atomic_store(p, v, __ATOMIC_RELAXED, __HIP_MEMORY_SCOPE_SYSTEM); }
__device__ __forceinline__ void st_agent(GAS u32* p, u32 v) { __hip_atomic_store(p, v, __ATOMIC_RELAXED, __HIP_MEMORY_SCOPE_AGENT); }

template <int OFF> __device__ __forceinline__ void st16_wt(const char* base, u32 voff, u4 v) {
  asm volatile("global_store_dwordx4 %0, %1, %2 offset:%3 sc0 sc1\n\ts_nop 1" :: "v"(voff), "v"(v), "s"(base), "n"(OFF) : "memory");
}
template <int OFF> __device__ __forceinline__ void ld16(f4& d, const char* base, u32 voff) {
  asm volatile("global_load_dwordx4 %0, %1, %2 offset:%3" : "=v"(d) : "v"(voff), "s"(base), "n"(OFF) : "memory");
}
template <int OFF> __device__ __forceinline__ void ld16_sys(f4& d, const char* base, u32 voff) {
  asm volatile("global_load_dwordx4 %0, %1, %2 offset:%3 sc0 sc1" : "=v"(d) : "v"(voff), "s"(base), "n"(OFF) : "memory");
}
__device__ __forceinline__ u4 cvt8(float a0, float a1, float a2, float a3, float a4, float a5, float a6, float a7) {
  h8 r; r[0] = (_Float16)a0; r[1] = (_Float16)a1; r[2] = (_Float16)a2; r[3] = (_Float16)a3;
  r[4] = (_Float16)a4; r[5] = (_Float16)a5; r[6] = (_Float16)a6; r[7] = (_Float16)a7;
  return __builtin_bit_cast(u4, r);
}
__device__ __forceinline__ u4 cvt8(const f4& a, const f4& b) { return cvt8(a[0], a[1], a[2], a[3], b[0], b[1], b[2], b[3]); }
__device__ __forceinline__ void pin4(f4* r) { asm volatile("" : "+v"(r[0]), "+v"(r[1]), "+v"(r[2]), "+v"(r[3]) :: "memory"); }

#define DBG_PUT(i, val) do { } while (0)
#define STAMP(i) DBG_PUT(i, __builtin_amdgcn_s_memrealtime())

struct Ctx {
  char* ws;
  GAS u32* tmo;
  u32 call;
  int wid, lane;
  size_t par;
};

__device__ __forceinline__ bool give_up(const Ctx& c, u64 t0) {
  if (__builtin_amdgcn_readfirstlane(ld_agent(c.tmo)) == c.call || __builtin_amdgcn_s_memrealtime() - t0 > DEADLINE) {
    st_agent(c.tmo, c.call); return true;
  }
  return false;
}

__device__ __forceinline__ u32 x_lane_off(int lane) { return (u32)((lane >> 2) * KDIM + 4 * ((lane & 3) ^ ((-(lane >> 4)) & 3))) * 4u; }
__device__ __forceinline__ const char* x_src(const float* x, int rb, int kt) { return (const char*)x + ((size_t)rb * 16 * KDIM + (size_t)kt * 64) * 4; }
__device__ __forceinline__ void x_load(f4* r, const char* s, u32 lo) { ld16<0>(r[0], s, lo); ld16<64>(r[1], s, lo); ld16<128>(r[2], s, lo); ld16<192>(r[3], s, lo); }
__device__ __forceinline__ u32 w_lane_off(int lane) { return (u32)(4 * (lane >> 4) * NLOC + 4 * (lane & 15)) * 4u; }
__device__ __forceinline__ const char* w_src(const float* w, int g, int kt) { return (const char*)w + ((size_t)kt * 64 * NLOC + (size_t)g * 64) * 4; }
__device__ __forceinline__ void w_load(f4* r, const char* s, u32 l0) {
  const u32 l1 = l0 + NLOC * 4, l2 = l0 + 2 * NLOC * 4, l3 = l0 + 3 * NLOC * 4;
#pragma unroll
  for (int i = 0; i < 4; ++i) { const char* t = s + (size_t)i * 16 * NLOC * 4;
    ld16<0>(r[4 * i], t, l0); ld16<0>(r[4 * i + 1], t, l1); ld16<0>(r[4 * i + 2], t, l2); ld16<0>(r[4 * i + 3], t, l3); }
}
__device__ __forceinline__ void w_cvt(u4* cv, const f4* r) {
#pragma unroll
  for (int j = 0; j < 4; ++j)
#pragma unroll
    for (int kk = 0; kk < 2; ++kk)
      cv[2 * j + kk] = cvt8(r[8 * kk][j], r[8 * kk + 1][j], r[8 * kk + 2][j], r[8 * kk + 3][j], r[8 * kk + 4][j], r[8 * kk + 5][j], r[8 * kk + 6][j], r[8 * kk + 7][j]);
}

__device__ __forceinline__ int in_qp(int u) { return u < 1024 ? 4 * (u >> 9) + (u & 3) : 8 + 2 * ((u - 1024) >> 8) + (u & 1); }
__device__ __forceinline__ int in_kt(int u) { return u < 1024 ? (u >> 2) & 127 : (u >> 1) & 127; }

struct Queue {
  int in, inn;
  int src;
  int cur, nxt;
  int ready;
  const char* ctr;
  u32 call;
  LAS int* slot;
  int it;
  u32 tk;
  __device__ __forceinline__ bool more_in() const { return in < UI || inn < UI || src < 2; }
  __device__ __forceinline__ void draw(bool out) {
    if (threadIdx.x < 2 && (threadIdx.x == 0 ? src < 2 : (out && nxt < UO))) {
      const u32 off = threadIdx.x == 0 ? src * 512 : 256, one = 1;
      asm volatile("global_atomic_add %0, %1, %2, %3 sc0" : "=v"(tk) : "v"(off), "v"(one), "s"(ctr) : "memory");
    }
  }
  __device__ __forceinline__ int post(bool out) {
    u32 v = 0;
    if (threadIdx.x < 2) {
      asm volatile("" : "+v"(tk));
      if (threadIdx.x == 0) {
        v = src == 0 ? tk - (call - 1u) * (u32)(UA - NC) + 2u * NC : tk - (call - 1u) * (u32)(UI - UA + NF) + UA;
        if (src == 2 || v >= (src == 0 ? UA : UI)) v = UI;
      } else {
        v = tk - (call - 1u) * (u32)(UO - NC) + 2u * NC;
        if (!(out && nxt < UO) || v >= UO) v = UO;
      }
      slot[(it & 1) * 4 + threadIdx.x] = (int)v;
    }
    return (int)v;
  }
  __device__ __forceinline__ void post_ready() { if (threadIdx.x == 0) slot[(it & 1) * 4 + 2] = ready; }
  __device__ __forceinline__ void advance(bool out) {
    in = inn; inn = __builtin_amdgcn_readfirstlane(slot[(it & 1) * 4]);
    if (inn == UI) src = (src == 0 && blockIdx.x < NF) ? 1 : 2;
    if (out) { cur = nxt; nxt = __builtin_amdgcn_readfirstlane(slot[(it & 1) * 4 + 1]); }
    ready = __builtin_amdgcn_readfirstlane(slot[(it & 1) * 4 + 2]);
    ++it;
  }
};

__device__ __forceinline__ void remote_tile(int t, int rank, int q, int& src, int& qp, int& c1) {
  const int og = (rank & 4) ^ 4, hp = t & 1, j = t >> 1;
  c1 = j & 1;
  if (j < 12) { const int r = j >> 1; if (r < 3) { src = rank; qp = 4 * r + 2 * hp; } else { src = rank ^ (r - 2); qp = 2 * hp; } }
  else if (j < 18) { src = rank ^ (((j - 12) >> 1) + 1); qp = 4 + 2 * hp; }
  else if (j < 24) { const int s = (j - 18) >> 1; src = og + s + (s >= q ? 1 : 0); qp = 12 + 2 * hp; }
  else { src = rank ^ (((j - 24) >> 1) + 1); qp = 8 + hp; }
  src &= 7;
}
__device__ __forceinline__ int unit_tile(int v) { return v < 384 ? v >> 3 : 48 + ((v - 384) >> 2); }
__device__ __forceinline__ int tile_unit(int t) { return t < 48 ? 8 * t : 384 + 4 * (t - 48); }

struct Task {
  const char* a;
  const char* b;
  const char* fa;
  const char* fb;
  const char* out;
  u32 ldo;
  char* done;
};

__device__ __forceinline__ float gelu(float y) {
  const float t = y * (y * y * 0.044715f + 1.0f) * -2.302208198f;
  return y * __builtin_amdgcn_rcpf(1.0f + __builtin_amdgcn_exp2f(t));
}

template <int OFF> __device__ __forceinline__ void glds(const char* g, u32 voff, u32 ldsdst) {
  asm volatile("s_mov_b32 m0, %2\n\ts_nop 0\n\tglobal_load_lds_dwordx4 %0, %1 offset:%3 sc0 sc1" :: "v"(voff), "s"(g), "s"(ldsdst), "n"(OFF) : "memory");
}

__device__ __forceinline__ int poll_ready(const Ctx& c, const char* fp, int ready, int need) {
  const u64 t0 = __builtin_amdgcn_s_memrealtime();
  for (;;) {
    const int g0 = ready >> 2;
    int g = g0 + (c.lane & 15); g = g < NKT / 4 ? g : NKT / 4 - 1;
    u4 f;
    asm volatile("global_load_dwordx4 %0, %1, off sc0 sc1\n\ts_waitcnt vmcnt(0)" : "=&v"(f) : "v"(fp + g * 16) : "memory");
    const int n = (int)(f[0] - c.call) < 0 ? 0 : (int)(f[1] - c.call) < 0 ? 1 : (int)(f[2] - c.call) < 0 ? 2 : (int)(f[3] - c.call) < 0 ? 3 : 4;
    const u64 m = __ballot(n == 4);
    const u32 all = (u32)(m & (m >> 16) & (m >> 32) & (m >> 48)) & 0xffffu;
    if (all == 0xffffu) ready = 4 * g0 + 64;
    else {
      const int i = __builtin_ctz(~all);
      const int a = __builtin_amdgcn_readlane(n, i), b = __builtin_amdgcn_readlane(n, i + 16), e = __builtin_amdgcn_readlane(n, i + 32), h = __builtin_amdgcn_readlane(n, i + 48);
      ready = 4 * (g0 + i) + min(min(a, b), min(e, h));
    }
    if (ready >= need) break;
    if (give_up(c, t0)) { ready = NKT; break; }
    __builtin_amdgcn_s_sleep(POLL_SLEEP);
  }
  return ready;
}

template <int TM>
__device__ __forceinline__ void gemm_task(const Ctx& c, LAS char* lds, const Task& t, int sb) {
  constexpr int AB = TM * 4096, ST = 32768, NL = TM == 4 ? 4 : 3;
  const int wid = c.wid, lane = c.lane;
  const int kh = wid >> 2, wm = (wid >> 1) & 1, wn = wid & 1, fr = lane & 15, fq = lane >> 4;
  const u32 lds0 = (u32)(uintptr_t)lds;
  const u32 voff = (u32)lane * 16u;
  const char* g0 = TM == 4 ? (wid < 4 ? t.a : t.b) + (wid & 3) * 4096 : t.b + wid * 2048;
  const char* g1 = t.a + wid * 1024;
  const u32 s0 = TM == 4 ? lds0 + wid * 4096 : lds0 + AB + wid * 2048, s1 = lds0 + wid * 1024;
  const char* fp = lane < 32 ? t.fa + (TM == 4 ? (lane >> 4) * 512 : 0) : t.fb + ((lane >> 4) & 1) * 512;
  const LAS char* pa = lds + wm * (TM * 2048) + kh * 1024 + (fr * 4 + (fq ^ ((-(fr >> 2)) & 3))) * 16;
  const LAS char* pb = lds + AB + wn * 8192 + kh * 1024 + lane * 16;
  f4 acc[TM][4];
#pragma unroll
  for (int m = 0; m < TM; ++m)
#pragma unroll
    for (int j = 0; j < 4; ++j) acc[m][j] = (f4){0.f, 0.f, 0.f, 0.f};
  h8 A0[TM], B0[4], A1[TM], B1[4];
  int ready = 0;
#define GL(i, kt, buf) do { if (!more) break; if (TM == 4) glds<(i) * 1024>(g0 + (size_t)(kt) * KT_B, voff, s0 + (buf) * ST); \
    else if ((i) < 2) glds<((i) & 1) * 1024>(g0 + (size_t)(kt) * KT_B, voff, s0 + (buf) * ST); \
    else if ((i) == 2) glds<0>(g1 + (size_t)(kt) * KT_B, voff, s1 + (buf) * ST); } while (0)
#define GLA(kt, buf) do { GL(0, kt, buf); GL(1, kt, buf); GL(2, kt, buf); GL(3, kt, buf); } while (0)
#define RA(A, m, buf) A[(m) % TM] = *(const LAS h8*)(pa + (buf) * ST + ((m) % TM) * 2048)
#define RB(B, j, buf) B[j] = *(const LAS h8*)(pb + (buf) * ST + (j) * 2048)
#define MF(A, B, m, j) acc[(m) % TM][j] = __builtin_amdgcn_mfma_f32_16x16x32_f16(A[(m) % TM], B[j], acc[(m) % TM][j], 0, 0, 0)
  if (wid == 0) ready = poll_ready(c, fp, 0, 4);
  raw_barrier(); if (wid == 0) STAMP(sb);
  { constexpr bool more = true; GLA(0, 0); GLA(1, 1); GLA(2, 2); GLA(3, 3); }
  wait_vm<3 * NL>(); raw_barrier();
#pragma unroll
  for (int m = 0; m < TM; ++m) RA(A0, m, 0);
#pragma unroll
  for (int j = 0; j < 4; ++j) RB(B0, j, 0);
#define STEP(s, buf, AC, BC, AN, BN, VM) do { \
    constexpr int nb = ((buf) + 1) & 3; \
    const int nx = (s) + 4; \
    if (more && wid == 0 && nx >= ready) { ready = poll_ready(c, fp, ready, nx + 1); STAMP(sb + 1); DBG_PUT(sb + 3, s); } \
    FENCE(); __builtin_amdgcn_s_waitcnt(waitcnt(VM, 0)); raw_barrier(); SB(); \
    if (TM == 4) { \
      MF(AC, BC, 0, 0); RA(AN, 0, nb); SB(); MF(AC, BC, 0, 1); RB(BN, 0, nb); SB(); MF(AC, BC, 0, 2); RA(AN, 1, nb); SB(); MF(AC, BC, 0, 3); RB(BN, 1, nb); SB(); \
      MF(AC, BC, 1, 0); RA(AN, 2, nb); SB(); MF(AC, BC, 1, 1); RB(BN, 2, nb); SB(); MF(AC, BC, 1, 2); RA(AN, 3, nb); SB(); MF(AC, BC, 1, 3); RB(BN, 3, nb); FENCE(); SB(); \
      MF(AC, BC, 2, 0); GL(0, nx, buf); SB(); MF(AC, BC, 2, 1); MF(AC, BC, 2, 2); GL(1, nx, buf); SB(); MF(AC, BC, 2, 3); MF(AC, BC, 3, 0); GL(2, nx, buf); SB(); \
      MF(AC, BC, 3, 1); MF(AC, BC, 3, 2); GL(3, nx, buf); SB(); MF(AC, BC, 3, 3); SB(); \
    } else { \
      MF(AC, BC, 0, 0); RA(AN, 0, nb); RB(BN, 0, nb); SB(); MF(AC, BC, 0, 1); RB(BN, 1, nb); RA(AN, 1, nb); SB(); MF(AC, BC, 0, 2); RB(BN, 2, nb); RB(BN, 3, nb); FENCE(); SB(); \
      MF(AC, BC, 0, 3); GL(0, nx, buf); SB(); MF(AC, BC, 1, 0); GL(1, nx, buf); SB(); MF(AC, BC, 1, 1); GL(2, nx, buf); SB(); MF(AC, BC, 1, 2); MF(AC, BC, 1, 3); SB(); \
    } } while (0)
  for (int s = 0; s < NKT - 4; s += 4) {
    constexpr bool more = true;
    STEP(s, 0, A0, B0, A1, B1, 2 * NL);
    STEP(s + 1, 1, A1, B1, A0, B0, 2 * NL);
    STEP(s + 2, 2, A0, B0, A1, B1, 2 * NL);
    STEP(s + 3, 3, A1, B1, A0, B0, 2 * NL);
  }
  u32 bad;
  { constexpr bool more = false; const u32 z = 0;
    asm volatile("global_load_dword %0, %1, %2 sc1" : "=v"(bad) : "v"(z), "s"(c.ws + TMO_OFF) : "memory");
    STEP(NKT - 4, 0, A0, B0, A1, B1, 2 * NL + 1);
    STEP(NKT - 3, 1, A1, B1, A0, B0, NL + 1);
    STEP(NKT - 2, 2, A0, B0, A1, B1, 0);
    asm volatile("" : "+v"(bad));
    STEP(NKT - 1, 3, A1, B1, A0, B0, 0);
  }
#undef STEP
#undef GL
#undef GLA
#undef RA
#undef RB
#undef MF
  if (wid == 0) STAMP(sb + 2);
  constexpr int H = TM / 2;
  LAS f4* ex = (LAS f4*)lds + wid * 512 + lane;
  LAS f4* ey = (LAS f4*)lds + (wid ^ 4) * 512 + lane;
  f4 v[H][4];
  if (kh == 0) {
#pragma unroll
    for (int m = 0; m < H; ++m)
#pragma unroll
      for (int j = 0; j < 4; ++j) { ex[(m * 4 + j) * 64] = acc[H + m][j]; v[m][j] = acc[m][j]; }
  } else {
#pragma unroll
    for (int m = 0; m < H; ++m)
#pragma unroll
      for (int j = 0; j < 4; ++j) { ex[(m * 4 + j) * 64] = acc[m][j]; v[m][j] = acc[H + m][j]; }
  }
  wait_lgkm(); raw_barrier();
#pragma unroll
  for (int m = 0; m < H; ++m)
#pragma unroll
    for (int j = 0; j < 4; ++j) v[m][j] += ey[(m * 4 + j) * 64];
  const bool poison = bad == c.call;
  const char* ob = t.out + (size_t)(wm * TM + kh * H) * 16 * t.ldo + wn * 256;
  const u32 oo = (u32)(4 * fq) * t.ldo + (u32)fr * 16u;
#pragma unroll
  for (int m = 0; m < H; ++m)
#pragma unroll
    for (int i = 0; i < 4; ++i) {
      f4 o = {gelu(v[m][0][i]), gelu(v[m][1][i]), gelu(v[m][2][i]), gelu(v[m][3][i])};
      if (poison) o = (f4){__builtin_nanf(""), __builtin_nanf(""), __builtin_nanf(""), __builtin_nanf("")};
      st16_wt<0>(ob + (size_t)(m * 16 + i) * t.ldo, oo, __builtin_bit_cast(u4, o));
    }
  if (t.done) wait_vm<0>();
  wait_lgkm(); raw_barrier();
  if (t.done && threadIdx.x == 0) st_sys((GAS u32*)t.done, c.call);
}

__device__ __forceinline__ u64 sel(bool c, u64 a, u64 b) { u64 m = c ? a : b; asm volatile("" : "+v"(m)); return m; }

__global__ __launch_bounds__(512, 2) void ag_gemm(Params p) {
  extern __shared__ __attribute__((aligned(1024))) char smem[];
  LAS char* lds = (LAS char*)smem;
  const int rank = p.rank & 7, me = rank & 3, og = (rank & 4) ^ 4;
  Ctx c;
  c.ws = p.ws[rank];
  c.wid = __builtin_amdgcn_readfirstlane(threadIdx.x >> 6); c.lane = threadIdx.x & 63;
  STAMP(0);
  const int xcd = blockIdx.x & 7, slot = blockIdx.x >> 3;
  c.tmo = (GAS u32*)(c.ws + TMO_OFF);
  auto get_call = [&]() {
    GAS u32* ctr = (GAS u32*)(c.ws + CTR_OFF + (size_t)blockIdx.x * 128);
    u32 call = (u32)__builtin_amdgcn_readfirstlane(ld_agent(ctr)) + 1u;
    asm volatile("" : "+s"(call) :: "memory");
    raw_barrier();
    if (threadIdx.x == 0) st_agent(ctr, call);
    c.call = call;
    c.par = DATA_OFF + (size_t)(call & 1u) * PAR_B;
  };

  if (slot < NCS) {
    if (c.wid >= 4) return;
    const u32 ld = (u32)c.lane * 16u, lx = x_lane_off(c.lane), lw = w_lane_off(c.lane);
    const int d = c.wid;
    f4 rx[4], rw[16];
#define IL(u) x_load(rx, x_src(p.x, 4 * in_qp(u) + c.wid, in_kt(u)), lx)
#define WL(u) w_load(rw, w_src(p.w, 4 * d + ((u) & 3), (u) >> 2), lw)
    IL((int)blockIdx.x); WL((int)blockIdx.x);
    get_call();
    STAMP(6);
    Queue q; q.in = q.cur = blockIdx.x; q.inn = q.nxt = blockIdx.x + NC; q.ready = 0; q.ctr = c.ws + CLM_OFF;
    q.src = 0; q.call = c.call;
    q.slot = (LAS int*)lds; q.it = 0;
    const size_t xo = c.par + (size_t)rank * STREAM_B + (size_t)c.wid * 2048;
    const char* i0 = p.ws[rank ^ 1] + xo; const char* i1 = p.ws[rank ^ 2] + xo; const char* i2 = p.ws[rank ^ 3] + xo;
    u64 pw = sel(c.lane == 1, (u64)p.ws[rank ^ 2], (u64)p.ws[rank ^ 1]); pw = sel(c.lane == 2, (u64)p.ws[rank ^ 3], pw);
    pw = sel(c.lane == 3, (u64)p.ws[og], pw); pw = sel(c.lane == 4, (u64)p.ws[og + 1], pw);
    pw = sel(c.lane == 5, (u64)p.ws[og + 2], pw); pw = sel(c.lane == 6, (u64)p.ws[og + 3], pw);
#define XOFF(qp, kt) ((size_t)(kt) * KT_B + (size_t)(qp) * 8192)
#define XFLAG(qp, kt) (u32)(FLG_OFF + rank * FLG_B + ((qp) * NKT + (kt)) * 4)
#define IN_CVT() const bool in_ = q.in < UI; const int iq_ = in_qp(q.in), ik_ = in_kt(q.in); \
      u4 x0 = cvt8(rx[0], rx[1]), x1 = cvt8(rx[2], rx[3]); asm volatile("" : "+v"(x0), "+v"(x1))
#define IN_LOAD() IL(q.inn < UI ? q.inn : 0)
#define IN_STORE() do { if (in_) { const size_t o_ = XOFF(iq_, ik_); \
      st16_wt<0>(i0 + o_, ld, x0); st16_wt<0>(i1 + o_, ld, x0); st16_wt<0>(i2 + o_, ld, x0); \
      st16_wt<1024>(i0 + o_, ld, x1); st16_wt<1024>(i1 + o_, ld, x1); st16_wt<1024>(i2 + o_, ld, x1); } } while (0)
#define IS(k) do { if (in_) st16_wt<((k) / 3) * 1024>(((k) % 3 == 0 ? i0 : (k) % 3 == 1 ? i1 : i2) + XOFF(iq_, ik_), ld, (k) < 3 ? x0 : x1); } while (0)
#define STORES(S) do { IS(0); S(0); IS(1); S(1); IS(2); S(2); S(3); IS(3); S(4); IS(4); S(5); IS(5); S(6); S(7); } while (0)
    int pin_ = -1;
#define PUBI(LAGN, out, oflag) do { if (LAGN <= 2) PUB(in_, XFLAG(iq_, ik_), out, oflag); \
      else { PUB(pin_ >= 0, pin_, out, oflag); pin_ = in_ ? (int)XFLAG(iq_, ik_) : -1; } } while (0)
#define PUB(in, fi, out, oflag) do { if (threadIdx.x < 7 && (c.lane < 3 ? (in) : (out))) \
      st_sys((GAS u32*)(pw + (u64)(c.lane < 3 ? (u32)(fi) : (u32)(oflag))), c.call); } while (0)
#define IN_ONLY(OFLAG, POLL) do { \
      IN_CVT(); q.draw(false); IN_LOAD(); IN_STORE(); \
      wait_vm<0>(); pin4(rx); q.post(false); POLL; q.post_ready(); wait_lgkm(); raw_barrier(); \
      PUB(in_, XFLAG(iq_, ik_), prev >= 0, OFLAG(prev)); prev = -1; \
      q.advance(false); } while (0)
#define PRODUCE(END, LAGN, LOADED, LOAD, PIN, CVT, ADDR, S, OFLAG, POLL, GATE) do { \
    if (q.cur < (END)) { \
      int prev = -1; \
      if (!(LOADED)) { GATE(q.cur); LOAD(q.cur); } \
      wait_vm<0>(); PIN(); pin4(rx); \
      while (q.nxt < (END)) { \
        GATE(q.nxt); \
        CVT(); IN_CVT(); q.draw(true); IN_LOAD(); LOAD(q.nxt); ADDR(q.cur); \
        STORES(S); \
        wait_vm<LAGN>(); if (LAGN > 8 && !in_) wait_vm<8>(); PIN(); pin4(rx); { const int v_ = q.post(true); POLL(v_); } q.post_ready(); wait_lgkm(); raw_barrier(); \
        PUBI(LAGN, prev >= 0, OFLAG(prev)); \
        STAMP(8 + (q.it < 55 ? q.it : 55)); \
        prev = q.cur; q.advance(true); \
      } \
      CVT(); IN_CVT(); q.draw(true); IN_LOAD(); ADDR(q.cur); \
      STORES(S); \
      wait_vm<LAGN>(); if (LAGN > 8 && !in_) wait_vm<8>(); pin4(rx); q.post(true); q.post_ready(); wait_lgkm(); raw_barrier(); \
      PUBI(LAGN, prev >= 0, OFLAG(prev)); \
      wait_vm<0>(); raw_barrier(); \
      PUB(pin_ >= 0, pin_, true, OFLAG(q.cur)); pin_ = -1; \
      q.advance(true); \
    } } while (0)
#define NOPOLL(v)
    {
      const char* wd = p.ws[og + d] + c.par + 9 * STREAM_B + (size_t)(4 * me) * 8192;
      u4 cv[8];
#define WP() do { pin4(rw); pin4(rw + 4); pin4(rw + 8); pin4(rw + 12); } while (0)
#define WC() do { w_cvt(cv, rw); _Pragma("unroll") for (int i = 0; i < 8; ++i) asm volatile("" : "+v"(cv[i])); } while (0)
#define WA(u) const char* e_ = wd + XOFF((u) & 3, (u) >> 2)
#define WS(i) st16_wt<((i) & 3) * 1024>(e_ + ((i) >> 2) * 4096, ld, cv[i])
#define WF(u) (FLG_OFF + 9 * FLG_B + ((4 * me + ((u) & 3)) * NKT + ((u) >> 2)) * 4)
      PRODUCE(UW, LAG_WY, true, WL, WP, WC, WA, WS, WF, NOPOLL, NOPOLL);
    }
    STAMP(1);
    {
      const char* o0 = p.ws[og] + xo; const char* o1 = p.ws[og + 1] + xo; const char* o2 = p.ws[og + 2] + xo; const char* o3 = p.ws[og + 3] + xo;
      f4 r[4]; u4 y0, y1;
#define YQ(u) (12 + ((u) & 3))
#define YK(u) (((u) >> 2) & 127)
#define YL(u) x_load(r, x_src(p.x, 4 * YQ(u) + c.wid, YK(u)), lx)
#define YP() pin4(r)
#define YC() do { y0 = cvt8(r[0], r[1]); y1 = cvt8(r[2], r[3]); asm volatile("" : "+v"(y0), "+v"(y1)); } while (0)
#define YA(u) const size_t e_ = XOFF(YQ(u), YK(u))
#define YS(i) st16_wt<((i) >> 2) * 1024>((((i) & 3) == 0 ? o0 : ((i) & 3) == 1 ? o1 : ((i) & 3) == 2 ? o2 : o3) + e_, ld, (i) < 4 ? y0 : y1)
#define YF(u) XFLAG(YQ(u), YK(u))
      PRODUCE(UX, LAG_WY, false, YL, YP, YC, YA, YS, YF, NOPOLL, NOPOLL);
    }
    STAMP(2);
    {
      const char* stg = c.ws + STG_OFF + (size_t)d * NSU * 8192;
      const char* ob = (const char*)p.out[og + d];
      const u32 lo = (u32)(c.lane >> 5) * (NLOC * 4) + (u32)(c.lane & 31) * 16u;
      const GAS u32* rf = (const GAS u32*)(c.ws + OST_OFF) + (c.lane < NST ? c.lane : NST - 1) * 4;
      f4 r[8]; u4 cv[8];
      auto look = [&]() {
        const u32 m0 = (ld_sys(rf) - c.call) | (ld_sys(rf + 1) - c.call) | (ld_sys(rf + 2) - c.call) | (ld_sys(rf + 3) - c.call);
        const u64 m = __ballot((int)m0 >= 0);
        q.ready = m == ~0ull ? 64 : __builtin_ctzll(~m);
      };
#define STAGED(u) (unit_tile((u) - UX) < q.ready)
#define OL(u) do { const char* s_ = stg + (size_t)((u) - UX) * 8192; const char* t_ = s_ + 4096; \
        ld16_sys<0>(r[0], s_, ld); ld16_sys<1024>(r[1], s_, ld); ld16_sys<2048>(r[2], s_, ld); ld16_sys<3072>(r[3], s_, ld); \
        ld16_sys<0>(r[4], t_, ld); ld16_sys<1024>(r[5], t_, ld); ld16_sys<2048>(r[6], t_, ld); ld16_sys<3072>(r[7], t_, ld); } while (0)
#define OP() do { pin4(r); pin4(r + 4); } while (0)
#define OC() do { _Pragma("unroll") for (int i = 0; i < 8; ++i) { cv[i] = __builtin_bit_cast(u4, r[i]); asm volatile("" : "+v"(cv[i])); } } while (0)
#define OA(u) int src_, qp_, c1_; const int t_ = unit_tile((u) - UX); remote_tile(t_, rank, d, src_, qp_, c1_); \
        const char* e_ = ob + ((size_t)(src_ * NLOC + qp_ * 64 + ((u) - UX - tile_unit(t_)) * 16) * NLOC + 256 * me + 128 * c1_) * 4
#define OS(i) st16_wt<0>(e_ + (size_t)(i) * 2 * NLOC * 4, lo, cv[i])
#define OF(u) (DON_OFF + rank * 2048 + ((u) - UX) * 4)
#define OPOLL(v) do { if (c.wid == 0) { const int u_ = __builtin_amdgcn_readlane(v, 1); if (u_ < UO && !STAGED(u_)) look(); } } while (0)
#define OGATE(u) do { \
        while (!STAGED(u) && q.more_in()) IN_ONLY(OF, if (c.wid == 0) look()); \
        if (!STAGED(u)) { \
          wait_vm<0>(); raw_barrier(); PUB(pin_ >= 0, pin_, prev >= 0, OF(prev)); prev = -1; pin_ = -1; \
          if (c.wid == 0) { const u64 t0 = __builtin_amdgcn_s_memrealtime(); \
            for (;;) { look(); if (STAGED(u)) break; if (give_up(c, t0)) { q.ready = 64; break; } __builtin_amdgcn_s_sleep(POLL_SLEEP); } } \
          q.post_ready(); wait_lgkm(); raw_barrier(); \
          q.ready = __builtin_amdgcn_readfirstlane(q.slot[(q.it & 1) * 4 + 2]); ++q.it; \
        } } while (0)
      if (q.cur < UO) {
        if (c.wid == 0) look();
        q.post_ready(); wait_lgkm(); raw_barrier();
        q.ready = __builtin_amdgcn_readfirstlane(q.slot[(q.it & 1) * 4 + 2]); ++q.it;
      }
      PRODUCE(UO, LAG_O, false, OL, OP, OC, OA, OS, OF, OPOLL, OGATE);
    }
    STAMP(3);
    { int prev = -1;
#define NOFLAG(u) 0
      while (q.more_in()) IN_ONLY(NOFLAG, (void)0); }
    STAMP(4);
    if (blockIdx.x == 0) {
      const GAS u32* dp = (const GAS u32*)(c.ws + DON_OFF + (og + c.wid) * 2048) + c.lane;
      const u64 t0 = __builtin_amdgcn_s_memrealtime();
      for (;;) {
        u32 m = ld_sys(dp + (c.lane < NSU + 12 - 384 ? 384 : 0)) - c.call;
#pragma unroll
        for (int i = 0; i < 6; ++i) m |= ld_sys(dp + 64 * i) - c.call;
        if (__all((int)m >= 0)) break;
        if (give_up(c, t0)) {
          if (c.lane == 0) ((GAS float*)p.out[rank])[(size_t)rank * NLOC * NLOC] = __builtin_nanf("");
          break;
        }
        __builtin_amdgcn_s_sleep(2);
      }
      STAMP(5);
    }
    return;
  }
  get_call();
  {
    const u32 ld = (u32)c.lane * 16u;
    if (blockIdx.x < 256 - NLOCAL) __builtin_amdgcn_s_sleep(1);
    else for (int i = blockIdx.x - (256 - NLOCAL); i < 512; i += NLOCAL) {
      const int kt0 = (i >> 4) * 4, half = i & 7;
      f4 r[16]; u4 cv[8];
      if (i & 8) {
        const int g = 2 * half + (c.wid & 1), kt = kt0 + (c.wid >> 1);
        w_load(r, w_src(p.w, g, kt), w_lane_off(c.lane));
        wait_vm<0>(); pin4(r); pin4(r + 4); pin4(r + 8); pin4(r + 12);
        w_cvt(cv, r);
        const char* dd = c.ws + c.par + 8 * STREAM_B + (size_t)kt * KT_B + (size_t)g * 8192;
        st16_wt<0>(dd, ld, cv[0]); st16_wt<1024>(dd, ld, cv[1]); st16_wt<2048>(dd, ld, cv[2]); st16_wt<3072>(dd, ld, cv[3]);
        st16_wt<0>(dd + 4096, ld, cv[4]); st16_wt<1024>(dd + 4096, ld, cv[5]); st16_wt<2048>(dd + 4096, ld, cv[6]); st16_wt<3072>(dd + 4096, ld, cv[7]);
      } else {
        const int rb = 8 * half + c.wid; const u32 ls = x_lane_off(c.lane);
        const char* s = x_src(p.x, rb, kt0);
        x_load(r, s, ls); x_load(r + 4, s + 256, ls); x_load(r + 8, s + 512, ls); x_load(r + 12, s + 768, ls);
        wait_vm<0>(); pin4(r); pin4(r + 4); pin4(r + 8); pin4(r + 12);
        const char* dd = c.ws + c.par + (size_t)rank * STREAM_B + (size_t)kt0 * KT_B + (size_t)rb * 2048;
#pragma unroll
        for (int k = 0; k < 4; ++k) { st16_wt<0>(dd + k * KT_B, ld, cvt8(r[4 * k], r[4 * k + 1])); st16_wt<1024>(dd + k * KT_B, ld, cvt8(r[4 * k + 2], r[4 * k + 3])); }
      }
      wait_vm<0>(); raw_barrier();
      if (threadIdx.x < 8) st_sys((GAS u32*)(c.ws + FLG_OFF + ((i & 8) ? 8 : rank) * FLG_B) + (2 * half + (threadIdx.x >> 2)) * NKT + kt0 + (threadIdx.x & 3), c.call);
    }
  }
  STAMP(1);
  const int li = slot - NCS;
  if (li >= NGS) return;
  const int hp = xcd & 1, q = xcd >> 1;
  auto make_task = [&](int src, int qp, int c1, int wstream) {
    const int ch = 2 * q + c1;
    Task t;
    t.a = c.ws + c.par + (size_t)src * STREAM_B + qp * 8192;
    t.b = c.ws + c.par + (size_t)wstream * STREAM_B + ch * 16384;
    t.fa = c.ws + FLG_OFF + src * FLG_B + qp * 512;
    t.fb = c.ws + FLG_OFF + wstream * FLG_B + ch * 1024;
    t.out = (const char*)(p.out[rank] + ((size_t)src * NLOC + qp * 64) * NLOC + 128 * ch); t.ldo = NLOC * 4; t.done = nullptr;
    return t;
  };
  auto stage = [&](Task& t, int rt) { t.out = c.ws + STG_OFF + (size_t)(q * NSU + tile_unit(rt)) * 8192; t.ldo = 512; t.done = c.ws + OST_OFF + (rt * 4 + q) * 4; };
  for (int ph = 0; ph < 2; ++ph) {
    int src, qp, c1, rt = -1;
    if (ph == 0) {
      if (li < 14) { const int r = li >> 1; c1 = li & 1; if (r < 4) { src = rank; qp = 4 * r + 2 * hp; } else { src = rank ^ (r - 3); qp = 2 * hp; } }
      else rt = 2 * (li - 14) + hp;
    } else if (li < 12) {
      const int a = li >> 2; c1 = li & 1; src = rank ^ (a + 1); qp = 4 + 2 * hp;
      if (li & 2) rt = 2 * (12 + a * 2 + c1) + hp;
    } else {
      const int j = li - 12;
      if (j < 8) { src = og + (j >> 1); c1 = j & 1; qp = 12 + 2 * hp; }
      else rt = 2 * (18 + j - 8) + hp;
    }
    if (rt >= 0) remote_tile(rt, rank, q, src, qp, c1);
    Task t = make_task(src & 7, qp, c1, rt >= 0 ? 9 : 8);
    if (rt >= 0) stage(t, rt);
    gemm_task<4>(c, lds, t, 8 + 8 * ph);
    STAMP(2 + ph);
  }
  if (li < 24) {
    const int last = li >= 12, j = li - 12 * last, a = j >> 2, c1 = j & 1, qp = 8 + 2 * last + hp, src = (rank ^ (a + 1)) & 7;
    const bool rem = (j & 2) != 0;
    Task t = make_task(src, qp, c1, rem ? 9 : 8);
    if (rem && !last) stage(t, 48 + a * 4 + c1 * 2 + hp);
    if (rem && last) {
      t.out = (const char*)(p.out[og + q] + ((size_t)src * NLOC + qp * 64) * NLOC + 256 * me + 128 * c1);
      t.done = p.ws[og + q] + DON_OFF + rank * 2048 + (NSU + a * 4 + c1 * 2 + hp) * 4;
    }
    gemm_task<2>(c, lds, t, 24);
    STAMP(4);
  }
}

}

extern "C" void kernel_launch(int rank, int nranks, ncclComm_t comm,
                              hipStream_t stream,
                              void* const* d_in, const int* in_sizes, int n_in,
                              void* d_out, int out_size,
                              const int* in_dtypes, int out_dtype,
                              void* const* d_out_all,
                              void* const* d_ws_all, size_t ws_bytes) {
  if (nranks != NR || n_in != 2 || ws_bytes < WS_NEED || (unsigned)rank >= (unsigned)NR ||
      in_sizes[0] != NLOC * KDIM || in_sizes[1] != KDIM * NLOC || out_size != KDIM * NLOC) {
    fprintf(stderr, "unexpected problem shape\n");
    return;
  }
  static bool attr = false;
  if (!attr) {
    HIP_CHECK(hipFuncSetAttribute((const void*)ag_gemm, hipFuncAttributeMaxDynamicSharedMemorySize, LDS_B));
    attr = true;
  }
  Params p;
  p.x = (const float*)d_in[0]; p.w = (const float*)d_in[1];
  for (int r = 0; r < NR; ++r) { p.ws[r] = (char*)d_ws_all[r]; p.out[r] = (float*)d_out_all[r]; }
  p.rank = rank; p.pad = 0;
  hipLaunchKernelGGL(ag_gemm, dim3(256), dim3(512), LDS_B, stream, p);
  HIP_CHECK(hipGetLastError());
}
```

```cpp
#include <hip/hip_runtime.h>
#include <rccl/rccl.h>
#include <cstdio>
#include <cstdint>

#define HIP_CHECK(expr) do { hipError_t _e = (expr); if (_e != hipSuccess) { \
    fprintf(stderr, "HIP error %s at %s:%d\n", hipGetErrorString(_e), __FILE__, __LINE__); \
    return; } } while (0)

typedef unsigned u32;
typedef unsigned long long u64;
typedef _Float16 h8 __attribute__((ext_vector_type(8)));
typedef float f4 __attribute__((ext_vector_type(4)));
typedef unsigned u4 __attribute__((ext_vector_type(4)));
typedef unsigned u2 __attribute__((ext_vector_type(2)));
#define GAS __attribute__((address_space(1)))
#define LAS __attribute__((address_space(3)))

#define NC_SLOTS 4
#define TAIL 56
#define PP 0
#define CVT_DELAY 2
#define NF 32
#define UB_UNITS 80
#define POLL_SLEEP 8

namespace {

constexpr int NR = 8;
constexpr int KDIM = 8192, NLOC = 1024;
constexpr int NKT = KDIM / 64;
constexpr int NC = 8 * NC_SLOTS, NCS = NC_SLOTS;
constexpr int NGS = 26;
static_assert(NCS + NGS <= 32, "grid");
#define N_LOCAL (256 - NC)
constexpr int NLOCAL = N_LOCAL;
constexpr int NST = 60;
constexpr int NSU = 48 * 8 + 12 * 4;
constexpr int UI = 1536;
constexpr int UW = 512, UX = 1024, UO = UX + NSU;
constexpr int UA = UI - UB_UNITS;
constexpr size_t CTR_OFF = 0;
constexpr size_t TMO_OFF = 32768;
constexpr size_t CLM_OFF = 32768 + 256;
constexpr size_t CVT_OFF = 34816;
constexpr size_t CVC_OFF = 45056;
constexpr size_t OST_OFF = 36864;
constexpr size_t DON_OFF = 40960;
constexpr size_t FLG_OFF = 65536;
constexpr size_t FLG_B = 8192;
constexpr size_t DBG_OFF = 262144;
constexpr size_t DATA_OFF = 1u << 20;
constexpr size_t KT_B = 131072;
constexpr size_t STREAM_B = (size_t)NKT * KT_B;
constexpr int NSTREAM = 10;
constexpr int PC_B = 1856;
constexpr size_t KTP_B = 64 * PC_B;
constexpr size_t PSTREAM_B = (size_t)NKT * KTP_B;
constexpr size_t PK_OFF = NSTREAM * STREAM_B;
constexpr size_t OVF_OFF = PK_OFF + 2 * PSTREAM_B;
constexpr size_t STG_OFF = OVF_OFF + (size_t)2 * NKT * 64 * 256;
constexpr size_t SOV_OFF = STG_OFF + (size_t)4 * NSU * 4 * PC_B;
constexpr size_t PAR_B = SOV_OFF + (size_t)4 * NSU * 4 * 256;
constexpr size_t WS_NEED = DATA_OFF + 2 * PAR_B;
constexpr int LDS_B = 131072;
constexpr u64 DEADLINE = 50000000ull;

struct Params {
  const float* x; const float* w;
  char* ws[NR]; float* out[NR];
  int rank; int pad;
};

#define FENCE() asm volatile("" ::: "memory")
#define SB() __builtin_amdgcn_sched_barrier(0)
__device__ __forceinline__ void raw_barrier() { FENCE(); __builtin_amdgcn_s_barrier(); FENCE(); }
template <int N> __device__ __forceinline__ void wait_vm() { asm volatile("s_waitcnt vmcnt(%0)" :: "n"(N) : "memory"); }
__device__ __forceinline__ void wait_lgkm() { FENCE(); __builtin_amdgcn_s_waitcnt(0xC07F); FENCE(); }
constexpr int waitcnt(int vm, int lgkm) { return (vm & 15) | (7 << 4) | ((lgkm & 15) << 8) | ((vm >> 4) << 14); }

__device__ __forceinline__ u32 ld_sys(const GAS u32* p) { return __hip_atomic_load(p, __ATOMIC_RELAXED, __HIP_MEMORY_SCOPE_SYSTEM); }
__device__ __forceinline__ u32 ld_agent(const GAS u32* p) { return __hip_atomic_load(p, __ATOMIC_RELAXED, __HIP_MEMORY_SCOPE_AGENT); }
__device__ __forceinline__ void st_sys(GAS u32* p, u32 v) { __hip_atomic_store(p, v, __ATOMIC_RELAXED, __HIP_MEMORY_SCOPE_SYSTEM); }
__device__ __forceinline__ void st_agent(GAS u32* p, u32 v) { __hip_atomic_store(p, v, __ATOMIC_RELAXED, __HIP_MEMORY_SCOPE_AGENT); }

template <int OFF> __device__ __forceinline__ void st16_wt(const char* base, u32 voff, u4 v) {
  asm volatile("global_store_dwordx4 %0, %1, %2 offset:%3 sc0 sc1\n\ts_nop 1" :: "v"(voff), "v"(v), "s"(base), "n"(OFF) : "memory");
}
template <int OFF> __device__ __forceinline__ void ld16(f4& d, const char* base, u32 voff) {
  asm volatile("global_load_dwordx4 %0, %1, %2 offset:%3" : "=v"(d) : "v"(voff), "s"(base), "n"(OFF) : "memory");
}
template <int OFF> __device__ __forceinline__ void ld16_sys(f4& d, const char* base, u32 voff) {
  asm volatile("global_load_dwordx4 %0, %1, %2 offset:%3 sc0 sc1" : "=v"(d) : "v"(voff), "s"(base), "n"(OFF) : "memory");
}
__device__ __forceinline__ u4 cvt8(float a0, float a1, float a2, float a3, float a4, float a5, float a6, float a7) {
  h8 r; r[0] = (_Float16)a0; r[1] = (_Float16)a1; r[2] = (_Float16)a2; r[3] = (_Float16)a3;
  r[4] = (_Float16)a4; r[5] = (_Float16)a5; r[6] = (_Float16)a6; r[7] = (_Float16)a7;
  return __builtin_bit_cast(u4, r);
}
__device__ __forceinline__ u4 cvt8(const f4& a, const f4& b) { return cvt8(a[0], a[1], a[2], a[3], b[0], b[1], b[2], b[3]); }
__device__ __forceinline__ void pin4(f4* r) { asm volatile("" : "+v"(r[0]), "+v"(r[1]), "+v"(r[2]), "+v"(r[3]) :: "memory"); }

#define DBG_PUT(i, val) do { } while (0)
#define STAMP(i) DBG_PUT(i, __builtin_amdgcn_s_memrealtime())

struct Ctx {
  char* ws;
  GAS u32* tmo;
  u32 call;
  int wid, lane;
  size_t par;
};

__device__ __forceinline__ bool give_up(const Ctx& c, u64 t0) {
  if (__builtin_amdgcn_readfirstlane(ld_agent(c.tmo)) == c.call || __builtin_amdgcn_s_memrealtime() - t0 > DEADLINE) {
    st_agent(c.tmo, c.call); return true;
  }
  return false;
}

__device__ __forceinline__ u32 x_lane_off(int lane) { return (u32)((lane >> 2) * KDIM + 4 * ((lane & 3) ^ ((-(lane >> 4)) & 3))) * 4u; }
__device__ __forceinline__ const char* x_src(const float* x, int rb, int kt) { return (const char*)x + ((size_t)rb * 16 * KDIM + (size_t)kt * 64) * 4; }
__device__ __forceinline__ void x_load(f4* r, const char* s, u32 lo) { ld16<0>(r[0], s, lo); ld16<64>(r[1], s, lo); ld16<128>(r[2], s, lo); ld16<192>(r[3], s, lo); }
__device__ __forceinline__ u32 w_lane_off(int lane) { return (u32)(4 * (lane >> 4) * NLOC + 4 * (lane & 15)) * 4u; }
__device__ __forceinline__ const char* w_src(const float* w, int g, int kt) { return (const char*)w + ((size_t)kt * 64 * NLOC + (size_t)g * 64) * 4; }
__device__ __forceinline__ void w_load(f4* r, const char* s, u32 l0) {
  const u32 l1 = l0 + NLOC * 4, l2 = l0 + 2 * NLOC * 4, l3 = l0 + 3 * NLOC * 4;
#pragma unroll
  for (int i = 0; i < 4; ++i) { const char* t = s + (size_t)i * 16 * NLOC * 4;
    ld16<0>(r[4 * i], t, l0); ld16<0>(r[4 * i + 1], t, l1); ld16<0>(r[4 * i + 2], t, l2); ld16<0>(r[4 * i + 3], t, l3); }
}
__device__ __forceinline__ void w_cvt(u4* cv, const f4* r) {
#pragma unroll
  for (int j = 0; j < 4; ++j)
#pragma unroll
    for (int kk = 0; kk < 2; ++kk)
      cv[2 * j + kk] = cvt8(r[8 * kk][j], r[8 * kk + 1][j], r[8 * kk + 2][j], r[8 * kk + 3][j], r[8 * kk + 4][j], r[8 * kk + 5][j], r[8 * kk + 6][j], r[8 * kk + 7][j]);
}

__device__ __forceinline__ u32 bfi(u32 m, u32 a, u32 b) { return (a & m) | (b & ~m); }
__device__ __forceinline__ u32 perm(u32 hi, u32 lo, u32 sel) { return __builtin_amdgcn_perm(hi, lo, sel); }
template <int CTRL, int RM> __device__ __forceinline__ u32 dpp_add(u32 v) { return v + (u32)__builtin_amdgcn_update_dpp(0, (int)v, CTRL, RM, 0xf, true); }
template <int CTRL, int RM> __device__ __forceinline__ u32 dpp_max(u32 v) { return max(v, (u32)__builtin_amdgcn_update_dpp(0, (int)v, CTRL, RM, 0xf, true)); }
__device__ __forceinline__ u32 wave_scan(u32 v) {
  v = dpp_add<0x111, 0xf>(v); v = dpp_add<0x112, 0xf>(v); v = dpp_add<0x114, 0xf>(v); v = dpp_add<0x118, 0xf>(v);
  v = dpp_add<0x142, 0xa>(v); v = dpp_add<0x143, 0xc>(v);
  return v;
}
template <int OFF> __device__ __forceinline__ void st8_wt(const char* base, u32 voff, u2 v) {
  asm volatile("global_store_dwordx2 %0, %1, %2 offset:%3 sc0 sc1" :: "v"(voff), "v"(v), "s"(base), "n"(OFF) : "memory");
}
template <int OFF> __device__ __forceinline__ void st4_wt(const char* base, u32 voff, u32 v) {
  asm volatile("global_store_dword %0, %1, %2 offset:%3 sc0 sc1" :: "v"(voff), "v"(v), "s"(base), "n"(OFF) : "memory");
}
__device__ __forceinline__ u32 esc_byte(u32 m, int lane) { const u32 b = __builtin_ctz(m); return (u32)lane * 16u + (b & 7u) * 4u + (b >> 3); }
template <bool F32>
__device__ __forceinline__ void pack_piece(u4 a, u4 b, LAS char* scr, const char* dst, const char* ovf, int lane) {
  constexpr int NH = F32 ? 2 : 4;
  constexpr u32 EM = F32 ? 0x7f7f7f7fu : 0x7c7c7c7cu, ONE = F32 ? 0x01010101u : 0x04040404u;
  u32 H[4] = {0, 0, 0, 0}, P[7];
  if (F32) {
    H[0] = perm(perm(a[3], a[2], 0x07030703u), perm(a[1], a[0], 0x07030703u), 0x05040100u);
    H[1] = perm(perm(b[3], b[2], 0x07030703u), perm(b[1], b[0], 0x07030703u), 0x05040100u);
    P[0] = perm(b[2], a[0], 0x04020100u); P[1] = perm(b[2], a[1], 0x05020100u); P[2] = perm(b[2], a[2], 0x06020100u);
    P[3] = perm(b[3], a[3], 0x04020100u); P[4] = perm(b[3], b[0], 0x05020100u); P[5] = perm(b[3], b[1], 0x06020100u);
  } else {
    H[0] = perm(a[1], a[0], 0x07050301u); H[1] = perm(a[3], a[2], 0x07050301u); H[2] = perm(b[1], b[0], 0x07050301u); H[3] = perm(b[3], b[2], 0x07050301u);
    P[0] = perm(a[1], a[0], 0x06040200u); P[1] = perm(a[3], a[2], 0x06040200u); P[2] = perm(b[1], b[0], 0x06040200u); P[3] = perm(b[3], b[2], 0x06040200u);
  }
  int lo;
  if (F32) {
    u32 x = 0;
#pragma unroll
    for (int k = 0; k < 8; ++k) x = max(x, (H[k & 1] >> (8 * (k >> 1))) & 0x7fu);
    x = dpp_max<0x111, 0xf>(x); x = dpp_max<0x112, 0xf>(x); x = dpp_max<0x114, 0xf>(x); x = dpp_max<0x118, 0xf>(x);
    x = dpp_max<0x142, 0xa>(x); x = dpp_max<0x143, 0xc>(x);
    lo = __builtin_amdgcn_readlane(x, 63) - 6;
  } else lo = (int)(((u32)__builtin_amdgcn_readlane(wave_scan(__builtin_amdgcn_sad_u8(H[0] & EM, 0u, 0u)), 63) + 2968u) >> 10) - 6;
  lo = lo < 0 ? 0 : lo > (F32 ? 120 : 25) ? (F32 ? 120 : 25) : lo;
  u32 m = 0, C[4];
#pragma unroll
  for (int j = 0; j < NH; ++j) {
    const u32 u = ((H[j] & EM) | 0x80808080u) - (u32)lo * ONE, df = u & 0x7f7f7f7fu;
    const u32 e = (~u | (df + (0x80808080u - 7u * ONE))) & 0x80808080u;
    const u32 cd = (df | ((e >> 7) * 0x7fu)) & (7u * ONE);
    C[j] = F32 ? cd | ((H[j] >> 4) & 0x08080808u) : bfi(EM, cd, H[j]);
    m |= e >> (7 - j);
  }
  if (F32) P[6] = C[0] | (C[1] << 4);
  else {
    const u32 z = (C[3] & 0x1f1f1f1fu) | ((C[3] >> 2) & 0x20202020u);
    P[4] = C[0] | ((z << 5) & 0x60606060u); P[5] = C[1] | ((z << 3) & 0x60606060u); P[6] = C[2] | ((z << 1) & 0x60606060u);
  }
  const u32 cnt = __builtin_popcount(m), inc = wave_scan(cnt);
  if ((u32)__builtin_amdgcn_readlane(inc, 63) > 63u) {
    P[0] = a[0]; P[1] = a[1]; P[2] = a[2]; P[3] = a[3]; P[4] = b[0]; P[5] = b[1]; P[6] = b[2];
    st4_wt<0>(ovf, (u32)lane * 4u, b[3]);
    lo = 255; m = 0;
  }
  *(LAS u4*)(scr + lane * 16) = (u4){H[0], H[1], H[2], H[3]};
  if (lane == 0) scr[1024] = (char)lo;
  u32 pos = inc - cnt + 1025u;
  while (m) { scr[pos++] = scr[esc_byte(m, lane)]; m &= m - 1; }
  st16_wt<0>(dst, (u32)lane * 16u, (u4){P[0], P[1], P[2], P[3]});
  st8_wt<1024>(dst, (u32)lane * 8u, (u2){P[4], P[5]});
  st4_wt<1536>(dst, (u32)lane * 4u, P[6]);
  if (lane < 16) st4_wt<1792>(dst, (u32)lane * 4u, *(const LAS u32*)(scr + 1024 + lane * 4));
}
template <bool F32>
__device__ __forceinline__ void unpack_piece(u4& a, u4& b, const LAS char* im, LAS char* scr, const char* ovf, int lane) {
  constexpr int NH = F32 ? 2 : 4;
  constexpr u32 ONE = F32 ? 0x01010101u : 0x04040404u;
  const u4 q = *(const LAS u4*)(im + lane * 16); const u2 r = *(const LAS u2*)(im + 1024 + lane * 8);
  const u32 t = *(const LAS u32*)(im + 1536 + lane * 4);
  const u32 lo = (u32)__builtin_amdgcn_readfirstlane(*(const LAS u32*)(im + 1792)) & 0xffu;
  if (lo == 255u) {
    u32 w;
    asm volatile("global_load_dword %0, %1, %2 sc0 sc1\n\ts_waitcnt vmcnt(0)" : "=&v"(w) : "v"((u32)lane * 4u), "s"(ovf) : "memory");
    a = q; b = (u4){r[0], r[1], t, w};
    return;
  }
  u32 H[4] = {r[0], r[1], t, 0}, m = 0;
  if (F32) { H[0] = t; H[1] = t >> 4; }
  else {
    const u32 z = ((r[0] & 0x60606060u) >> 5) | ((r[1] & 0x60606060u) >> 3) | ((t & 0x60606060u) >> 1);
    H[3] = (z & 0x1f1f1f1fu) | ((z << 2) & 0x80808080u);
  }
#pragma unroll
  for (int j = 0; j < NH; ++j) {
    const u32 cd = H[j] & (7u * ONE);
    H[j] = F32 ? (cd + lo * ONE) | ((H[j] & 0x08080808u) << 4) : bfi(0x7c7c7c7cu, cd + lo * ONE, H[j]);
    m |= ((cd + ONE) & (8u * ONE)) >> ((F32 ? 3 : 5) - j);
  }
  if (!F32 || __any(m != 0)) {
    *(LAS u4*)(scr + lane * 16) = (u4){H[0], H[1], H[2], H[3]};
    const u32 cnt = __builtin_popcount(m);
    u32 pos = wave_scan(cnt) - cnt + 1793u;
    while (m) {
      const u32 w0 = *(const LAS u32*)(im + (pos & ~3u)), w1 = *(const LAS u32*)(im + (pos & ~3u) + 4);
      u32 w = __builtin_amdgcn_alignbyte(w1, w0, pos & 3u);
#pragma unroll
      for (int k = 0; k < 4; ++k) if (m) { scr[esc_byte(m, lane)] = (char)w; m &= m - 1; w >>= 8; }
      pos += 4;
    }
    const u4 h = *(const LAS u4*)(scr + lane * 16);
    H[0] = h[0]; H[1] = h[1]; H[2] = h[2]; H[3] = h[3];
  }
  if (F32) {
    a = (u4){perm(H[0], q[0], 0x04020100u), perm(H[0], q[1], 0x05020100u), perm(H[0], q[2], 0x06020100u), perm(H[0], q[3], 0x07020100u)};
    b = (u4){perm(H[1], r[0], 0x04020100u), perm(H[1], r[1], 0x05020100u),
             perm(perm(H[1], q[2], 0x06030603u), perm(q[1], q[0], 0x07030703u), 0x05040100u),
             perm(perm(H[1], r[1], 0x07030703u), perm(r[0], q[3], 0x07030703u), 0x05040100u)};
  } else {
    a = (u4){perm(H[0], q[0], 0x05010400u), perm(H[0], q[0], 0x07030602u), perm(H[1], q[1], 0x05010400u), perm(H[1], q[1], 0x07030602u)};
    b = (u4){perm(H[2], q[2], 0x05010400u), perm(H[2], q[2], 0x07030602u), perm(H[3], q[3], 0x05010400u), perm(H[3], q[3], 0x07030602u)};
  }
}

__device__ __forceinline__ int late_before(int s) { return s < 256 - TAIL ? s * (128 - TAIL) / (256 - TAIL) : s - 128; }
__device__ __forceinline__ int in_qp(int u) {
  if (u < 1024) return 4 * (u >> 9) + (u & 3);
  const int s = (u - 1024) >> 1;
  return (late_before(s + 1) > late_before(s) ? 10 : 8) + (u & 1);
}
__device__ __forceinline__ int in_kt(int u) {
  if (u < 1024) return (u >> 2) & 127;
  const int s = (u - 1024) >> 1, n = late_before(s);
  return (late_before(s + 1) > n ? n : s - n) & 127;
}

__device__ __forceinline__ void remote_tile(int t, int rank, int q, int& src, int& qp, int& c1) {
  const int og = (rank & 4) ^ 4, hp = t & 1, j = t >> 1;
  c1 = j & 1;
  if (j < 12) { const int r = j >> 1; if (r < 3) { src = rank; qp = 4 * r + 2 * hp; } else { src = rank ^ (r - 2); qp = 2 * hp; } }
  else if (j < 18) { src = rank ^ (((j - 12) >> 1) + 1); qp = 4 + 2 * hp; }
  else if (j < 24) { const int s = (j - 18) >> 1; src = og + s + (s >= q ? 1 : 0); qp = 12 + 2 * hp; }
  else { src = rank ^ (((j - 24) >> 1) + 1); qp = 8 + hp; }
  src &= 7;
}
__device__ __forceinline__ int unit_tile(int v) { return v < 384 ? v >> 3 : 48 + ((v - 384) >> 2); }
__device__ __forceinline__ int tile_unit(int t) { return t < 48 ? 8 * t : 384 + 4 * (t - 48); }

struct Task {
  const char* a;
  const char* b;
  const char* fa;
  const char* fb;
  const char* out;
  const char* ovf;
  char* done;
};

__device__ __forceinline__ float gelu(float y) {
  const float t = y * (y * y * 0.044715f + 1.0f) * -2.302208198f;
  return y * __builtin_amdgcn_rcpf(1.0f + __builtin_amdgcn_exp2f(t));
}

template <int OFF> __device__ __forceinline__ void glds(const char* g, u32 voff, u32 ldsdst) {
  asm volatile("s_mov_b32 m0, %2\n\ts_nop 0\n\tglobal_load_lds_dwordx4 %0, %1 offset:%3 sc0 sc1" :: "v"(voff), "s"(g), "s"(ldsdst), "n"(OFF) : "memory");
}

__device__ __forceinline__ int poll_ready(const Ctx& c, const char* fp, int ready, int need) {
  const u64 t0 = __builtin_amdgcn_s_memrealtime();
  for (;;) {
    const int g0 = ready >> 2;
    int g = g0 + (c.lane & 15); g = g < NKT / 4 ? g : NKT / 4 - 1;
    u4 f;
    asm volatile("global_load_dwordx4 %0, %1, off sc0 sc1\n\ts_waitcnt vmcnt(0)" : "=&v"(f) : "v"(fp + g * 16) : "memory");
    const int n = (int)(f[0] - c.call) < 0 ? 0 : (int)(f[1] - c.call) < 0 ? 1 : (int)(f[2] - c.call) < 0 ? 2 : (int)(f[3] - c.call) < 0 ? 3 : 4;
    const u64 m = __ballot(n == 4);
    const u32 all = (u32)(m & (m >> 16) & (m >> 32) & (m >> 48)) & 0xffffu;
    if (all == 0xffffu) ready = 4 * g0 + 64;
    else {
      const int i = __builtin_ctz(~all);
      const int a = __builtin_amdgcn_readlane(n, i), b = __builtin_amdgcn_readlane(n, i + 16), e = __builtin_amdgcn_readlane(n, i + 32), h = __builtin_amdgcn_readlane(n, i + 48);
      ready = 4 * (g0 + i) + min(min(a, b), min(e, h));
    }
    if (ready >= need) break;
    if (give_up(c, t0)) { ready = NKT; break; }
    __builtin_amdgcn_s_sleep(POLL_SLEEP);
  }
  return ready;
}

template <int TM>
__device__ __forceinline__ void gemm_task(const Ctx& c, LAS char* lds, const Task& t, int sb) {
  constexpr int AB = TM * 4096, ST = 32768, NL = TM == 4 ? 4 : 3;
  const int wid = c.wid, lane = c.lane;
  const int kh = wid >> 2, wm = (wid >> 1) & 1, wn = wid & 1, fr = lane & 15, fq = lane >> 4;
  const u32 lds0 = (u32)(uintptr_t)lds;
  const u32 voff = (u32)lane * 16u;
  const char* g0 = TM == 4 ? (wid < 4 ? t.a : t.b) + (wid & 3) * 4096 : t.b + wid * 2048;
  const char* g1 = t.a + wid * 1024;
  const u32 s0 = TM == 4 ? lds0 + wid * 4096 : lds0 + AB + wid * 2048, s1 = lds0 + wid * 1024;
  const char* fp = lane < 32 ? t.fa + (TM == 4 ? (lane >> 4) * 512 : 0) : t.fb + ((lane >> 4) & 1) * 512;
  const LAS char* pa = lds + wm * (TM * 2048) + kh * 1024 + (fr * 4 + (fq ^ ((-(fr >> 2)) & 3))) * 16;
  const LAS char* pb = lds + AB + wn * 8192 + kh * 1024 + lane * 16;
  f4 acc[TM][4];
#pragma unroll
  for (int m = 0; m < TM; ++m)
#pragma unroll
    for (int j = 0; j < 4; ++j) acc[m][j] = (f4){0.f, 0.f, 0.f, 0.f};
  h8 A0[TM], B0[4], A1[TM], B1[4];
  int ready = 0;
#define GL(i, kt, buf) do { if (!more) break; if (TM == 4) glds<(i) * 1024>(g0 + (size_t)(kt) * KT_B, voff, s0 + (buf) * ST); \
    else if ((i) < 2) glds<((i) & 1) * 1024>(g0 + (size_t)(kt) * KT_B, voff, s0 + (buf) * ST); \
    else if ((i) == 2) glds<0>(g1 + (size_t)(kt) * KT_B, voff, s1 + (buf) * ST); } while (0)
#define GLA(kt, buf) do { GL(0, kt, buf); GL(1, kt, buf); GL(2, kt, buf); GL(3, kt, buf); } while (0)
#define RA(A, m, buf) A[(m) % TM] = *(const LAS h8*)(pa + (buf) * ST + ((m) % TM) * 2048)
#define RB(B, j, buf) B[j] = *(const LAS h8*)(pb + (buf) * ST + (j) * 2048)
#define MF(A, B, m, j) acc[(m) % TM][j] = __builtin_amdgcn_mfma_f32_16x16x32_f16(A[(m) % TM], B[j], acc[(m) % TM][j], 0, 0, 0)
  if (wid == 0) ready = poll_ready(c, fp, 0, 4);
  raw_barrier(); if (wid == 0) STAMP(sb);
  { constexpr bool more = true; GLA(0, 0); GLA(1, 1); GLA(2, 2); GLA(3, 3); }
  wait_vm<3 * NL>(); raw_barrier();
#pragma unroll
  for (int m = 0; m < TM; ++m) RA(A0, m, 0);
#pragma unroll
  for (int j = 0; j < 4; ++j) RB(B0, j, 0);
#define STEP(s, buf, AC, BC, AN, BN, VM) do { \
    constexpr int nb = ((buf) + 1) & 3; \
    const int nx = (s) + 4; \
    if (more && wid == 0 && nx >= ready) { ready = poll_ready(c, fp, ready, nx + 1); STAMP(sb + 1); DBG_PUT(sb + 3, s); } \
    FENCE(); __builtin_amdgcn_s_waitcnt(waitcnt(VM, 0)); raw_barrier(); SB(); \
    if (TM == 4) { \
      MF(AC, BC, 0, 0); RA(AN, 0, nb); SB(); MF(AC, BC, 0, 1); RB(BN, 0, nb); SB(); MF(AC, BC, 0, 2); RA(AN, 1, nb); SB(); MF(AC, BC, 0, 3); RB(BN, 1, nb); SB(); \
      MF(AC, BC, 1, 0); RA(AN, 2, nb); SB(); MF(AC, BC, 1, 1); RB(BN, 2, nb); SB(); MF(AC, BC, 1, 2); RA(AN, 3, nb); SB(); MF(AC, BC, 1, 3); RB(BN, 3, nb); FENCE(); SB(); \
      MF(AC, BC, 2, 0); GL(0, nx, buf); SB(); MF(AC, BC, 2, 1); MF(AC, BC, 2, 2); GL(1, nx, buf); SB(); MF(AC, BC, 2, 3); MF(AC, BC, 3, 0); GL(2, nx, buf); SB(); \
      MF(AC, BC, 3, 1); MF(AC, BC, 3, 2); GL(3, nx, buf); SB(); MF(AC, BC, 3, 3); SB(); \
    } else { \
      MF(AC, BC, 0, 0); RA(AN, 0, nb); RB(BN, 0, nb); SB(); MF(AC, BC, 0, 1); RB(BN, 1, nb); RA(AN, 1, nb); SB(); MF(AC, BC, 0, 2); RB(BN, 2, nb); RB(BN, 3, nb); FENCE(); SB(); \
      MF(AC, BC, 0, 3); GL(0, nx, buf); SB(); MF(AC, BC, 1, 0); GL(1, nx, buf); SB(); MF(AC, BC, 1, 1); GL(2, nx, buf); SB(); MF(AC, BC, 1, 2); MF(AC, BC, 1, 3); SB(); \
    } } while (0)
  for (int s = 0; s < NKT - 4; s += 4) {
    constexpr bool more = true;
    STEP(s, 0, A0, B0, A1, B1, 2 * NL);
    STEP(s + 1, 1, A1, B1, A0, B0, 2 * NL);
    STEP(s + 2, 2, A0, B0, A1, B1, 2 * NL);
    STEP(s + 3, 3, A1, B1, A0, B0, 2 * NL);
  }
  u32 bad;
  { constexpr bool more = false; const u32 z = 0;
    asm volatile("global_load_dword %0, %1, %2 sc1" : "=v"(bad) : "v"(z), "s"(c.ws + TMO_OFF) : "memory");
    STEP(NKT - 4, 0, A0, B0, A1, B1, 2 * NL + 1);
    STEP(NKT - 3, 1, A1, B1, A0, B0, NL + 1);
    STEP(NKT - 2, 2, A0, B0, A1, B1, 0);
    asm volatile("" : "+v"(bad));
    STEP(NKT - 1, 3, A1, B1, A0, B0, 0);
  }
#undef STEP
#undef GL
#undef GLA
#undef RA
#undef RB
#undef MF
  if (wid == 0) STAMP(sb + 2);
  constexpr int H = TM / 2;
  LAS f4* ex = (LAS f4*)lds + wid * 512 + lane;
  LAS f4* ey = (LAS f4*)lds + (wid ^ 4) * 512 + lane;
  f4 v[H][4];
  if (kh == 0) {
#pragma unroll
    for (int m = 0; m < H; ++m)
#pragma unroll
      for (int j = 0; j < 4; ++j) { ex[(m * 4 + j) * 64] = acc[H + m][j]; v[m][j] = acc[m][j]; }
  } else {
#pragma unroll
    for (int m = 0; m < H; ++m)
#pragma unroll
      for (int j = 0; j < 4; ++j) { ex[(m * 4 + j) * 64] = acc[m][j]; v[m][j] = acc[H + m][j]; }
  }
  wait_lgkm(); raw_barrier();
#pragma unroll
  for (int m = 0; m < H; ++m)
#pragma unroll
    for (int j = 0; j < 4; ++j) v[m][j] += ey[(m * 4 + j) * 64];
  const bool poison = bad == c.call;
  const char* ob = t.out + (size_t)(wm * TM + kh * H) * 16 * (NLOC * 4) + wn * 256;
  const u32 oo = (u32)(4 * fq) * (NLOC * 4) + (u32)fr * 16u;
#pragma unroll
  for (int m = 0; m < H; ++m) {
    u4 o[4];
#pragma unroll
    for (int i = 0; i < 4; ++i) {
      f4 g = {gelu(v[m][0][i]), gelu(v[m][1][i]), gelu(v[m][2][i]), gelu(v[m][3][i])};
      if (poison) g = (f4){__builtin_nanf(""), __builtin_nanf(""), __builtin_nanf(""), __builtin_nanf("")};
      o[i] = __builtin_bit_cast(u4, g);
    }
    if (t.ovf) {
      const int pc = (wm * TM + kh * H + m) * 4 + 2 * wn;
      pack_piece<true>(o[0], o[1], lds + 65536 + wid * 2048, t.out + pc * PC_B, t.ovf + pc * 256, lane);
      pack_piece<true>(o[2], o[3], lds + 65536 + wid * 2048, t.out + (pc + 1) * PC_B, t.ovf + (pc + 1) * 256, lane);
    } else {
#pragma unroll
      for (int i = 0; i < 4; ++i) st16_wt<0>(ob + (size_t)(m * 16 + i) * (NLOC * 4), oo, o[i]);
    }
  }
  if (t.done) wait_vm<0>();
  wait_lgkm(); raw_barrier();
  if (t.done && threadIdx.x == 0) st_sys((GAS u32*)t.done, c.call);
}

__global__ __launch_bounds__(512, 2) void ag_gemm(Params p) {
  extern __shared__ __attribute__((aligned(1024))) char smem[];
  LAS char* lds = (LAS char*)smem;
  const int rank = p.rank & 7, me = rank & 3, og = (rank & 4) ^ 4;
  Ctx c;
  c.ws = p.ws[rank];
  c.wid = __builtin_amdgcn_readfirstlane(threadIdx.x >> 6); c.lane = threadIdx.x & 63;
  STAMP(0);
  const int xcd = blockIdx.x & 7, slot = blockIdx.x >> 3;
  c.tmo = (GAS u32*)(c.ws + TMO_OFF);
  auto get_call = [&]() {
    GAS u32* ctr = (GAS u32*)(c.ws + CTR_OFF + (size_t)blockIdx.x * 128);
    u32 call = (u32)__builtin_amdgcn_readfirstlane(ld_agent(ctr)) + 1u;
    asm volatile("" : "+s"(call) :: "memory");
    raw_barrier();
    if (threadIdx.x == 0) st_agent(ctr, call);
    c.call = call;
    c.par = DATA_OFF + (size_t)(call & 1u) * PAR_B;
  };

  if (slot < NCS) {
    const u32 ld = (u32)c.lane * 16u;
#define XOFF(qp, kt) ((size_t)(kt) * KT_B + (size_t)(qp) * 8192)
#pragma unroll
    for (int k = 0; k < PP; ++k) {
      const int u = blockIdx.x + k * NC, d = c.wid & 3;
      f4 rx[4], rw[16]; u4 cv[8];
      if (c.wid < 4) {
        x_load(rx, x_src(p.x, 4 * (u & 3) + d, u >> 2), x_lane_off(c.lane));
        w_load(rw, w_src(p.w, 4 * d + (u & 3), u >> 2), w_lane_off(c.lane));
      }
      if (k == 0) get_call();
      if (c.wid < 4) {
        wait_vm<0>(); pin4(rx); pin4(rw); pin4(rw + 4); pin4(rw + 8); pin4(rw + 12);
        const u4 x0 = cvt8(rx[0], rx[1]), x1 = cvt8(rx[2], rx[3]); w_cvt(cv, rw);
        const size_t xo = c.par + (size_t)rank * STREAM_B + XOFF(u & 3, u >> 2) + d * 2048;
        const char* e0 = p.ws[rank ^ 1] + xo; const char* e1 = p.ws[rank ^ 2] + xo; const char* e2 = p.ws[rank ^ 3] + xo;
        const char* eo = p.ws[og + d] + c.par + 9 * STREAM_B + XOFF(4 * me + (u & 3), u >> 2); const char* ep = eo + 4096;
        st16_wt<0>(e0, ld, x0); st16_wt<0>(eo, ld, cv[0]); st16_wt<0>(e1, ld, x0); st16_wt<1024>(eo, ld, cv[1]);
        st16_wt<0>(e2, ld, x0); st16_wt<2048>(eo, ld, cv[2]); st16_wt<3072>(eo, ld, cv[3]);
        st16_wt<1024>(e0, ld, x1); st16_wt<0>(ep, ld, cv[4]); st16_wt<1024>(e1, ld, x1); st16_wt<1024>(ep, ld, cv[5]);
        st16_wt<1024>(e2, ld, x1); st16_wt<2048>(ep, ld, cv[6]); st16_wt<3072>(ep, ld, cv[7]);
      }
      wait_vm<0>(); raw_barrier();
      if (threadIdx.x < 7) {
        const char* pw = p.ws[(c.lane < 3 ? rank ^ (c.lane + 1) : og + c.lane - 3) & 7];
        st_sys((GAS u32*)(pw + FLG_OFF + (c.lane < 3 ? rank * FLG_B + ((u & 3) * NKT + (u >> 2)) * 4
                                                     : 9 * FLG_B + ((4 * me + (u & 3)) * NKT + (u >> 2)) * 4)), c.call);
      }
    }
    if (PP == 0) get_call();
    STAMP(6);
    LAS int* post = (LAS int*)(lds + LDS_B - 64);
    if (c.wid == 7) {
      const GAS u32* cvp = (const GAS u32*)(c.ws + CVT_OFF + (rank ^ ((c.lane < 56 ? c.lane >> 3 : 6) + 1)) * 128);
      const GAS u32* stp = (const GAS u32*)(c.ws + OST_OFF + (c.lane < NST ? c.lane : NST - 1) * 16);
      int inn = blockIdx.x + PP * NC, src = 0, nxt = blockIdx.x + PP * NC;
      int in1 = UI, out1 = UO, in2 = UI, out2 = UO;
      int cvt = 0, ready = 0, dead = 0;
      auto look = [&]() {
        if (cvt < 32) {
          const int g = cvt + (c.lane & 7);
          u64 m = __ballot((int)(ld_sys(cvp + (g < 32 ? g : 31)) - c.call) >= 0);
          m &= m >> 32; m &= m >> 16; m &= m >> 8;
          const u32 a = (u32)m & 0xffu;
          cvt += a == 0xffu ? 8 : __builtin_ctz(~a);
        }
        if (nxt >= UX && ready < NST) {
          const u64 m = __ballot((int)((ld_sys(stp) - c.call) | (ld_sys(stp + 1) - c.call) | (ld_sys(stp + 2) - c.call) | (ld_sys(stp + 3) - c.call)) >= 0);
          ready = m == ~0ull ? 64 : __builtin_ctzll(~m);
        }
      };
#define AVAIL_IN(u) ((u) < UI && (in_kt(u) >> 2) < cvt)
#define AVAIL_OUT(u) ((u) < UO && ((u) < UX ? (((u) >> 4) & 31) < cvt : unit_tile((u) - UX) < ready))
      for (int it = 0;; ++it) {
        if ((inn < UI && !AVAIL_IN(inn)) || (nxt < UO && !AVAIL_OUT(nxt))) look();
        const bool busy = in1 < UI || out1 < UO || in2 < UI || out2 < UO;
        if (!busy && (inn < UI || nxt < UO)) {
          const u64 t0 = __builtin_amdgcn_s_memrealtime();
          while (!AVAIL_IN(inn) && !AVAIL_OUT(nxt)) {
            if (give_up(c, t0)) { cvt = 32; ready = 64; dead = 2; break; }
            __builtin_amdgcn_s_sleep(1);
            look();
          }
        }
        const bool take_in = AVAIL_IN(inn), take_out = AVAIL_OUT(nxt);
        const int in0 = take_in ? inn : UI, out0 = take_out ? nxt : UO;
        const bool stop = !busy && !take_in && !take_out && src == 2;
        if (c.lane < 3) post[(it & 1) * 4 + c.lane] = c.lane == 0 ? in0 : c.lane == 1 ? out0 : (stop ? 1 : dead);
        wait_lgkm(); raw_barrier();
        if (c.lane < 7 && (c.lane < 3 ? in2 < UI : out2 < UX)) {
          const u32 fo = c.lane < 3 ? (u32)(FLG_OFF + (rank ^ (c.lane + 1)) * FLG_B + (in_qp(in2) * NKT + in_kt(in2)) * 4)
                       : out2 < UW ? (u32)(FLG_OFF + 9 * FLG_B + ((4 * (c.lane - 3) + (out2 & 3)) * NKT + (out2 >> 2)) * 4)
                                   : (u32)(FLG_OFF + (og + c.lane - 3) * FLG_B + ((12 + (out2 & 3)) * NKT + ((out2 >> 2) & 127)) * 4);
          st_sys((GAS u32*)(c.ws + fo), c.call);
        }
        STAMP(8 + (it < 30 ? it : 30));
        if (stop) break;
        in2 = in1; out2 = out1; in1 = in0; out1 = out0;
        const bool draw_in = src < 2 && (inn == UI || take_in);
        u32 tk = 0;
        if (c.lane < 2 && (c.lane == 0 ? draw_in : take_out))
          tk = __hip_atomic_fetch_add((GAS u32*)(c.ws + CLM_OFF + (c.lane == 0 ? src * 512 : 256)), 1u, __ATOMIC_RELAXED, __HIP_MEMORY_SCOPE_AGENT);
        if (draw_in) {
          const u32 t = __builtin_amdgcn_readlane(tk, 0);
          const u32 v = src == 0 ? t - (c.call - 1u) * (u32)(UA - PP * NC) + (PP + 1) * NC : t - (c.call - 1u) * (u32)(UI - UA + NF) + UA;
          inn = v < (src == 0 ? UA : UI) ? (int)v : UI;
          if (inn == UI) src = (src == 0 && blockIdx.x < NF) ? 1 : 2;
        }
        if (take_out) {
          const u32 v = __builtin_amdgcn_readlane(tk, 1) - (c.call - 1u) * (u32)(UO - PP * NC) + (PP + 1) * NC;
          nxt = v < UO ? (int)v : UO;
        }
      }
      STAMP(4);
      if (blockIdx.x == 0) {
        const GAS u32* dp = (const GAS u32*)(c.ws + DON_OFF) + ((c.lane & 15) < 12 ? c.lane : (c.lane & 48));
        const u64 t0 = __builtin_amdgcn_s_memrealtime();
        for (;;) {
          if (__all((int)(ld_sys(dp) - c.call) >= 0)) break;
          if (give_up(c, t0)) {
            if (c.lane == 0) ((GAS float*)p.out[rank])[(size_t)rank * NLOC * NLOC] = __builtin_nanf("");
            break;
          }
          __builtin_amdgcn_s_sleep(1);
        }
        STAMP(5);
      }
      return;
    }
    const bool ing = c.wid < 3;
    const int d = c.wid - 3, peer = (ing ? rank ^ (c.wid + 1) : og + d) & 7;
    const char* pw = p.ws[peer] + c.par;
    LAS char* img = lds + c.wid * 17408;
    const u32 lo = (u32)(c.lane >> 4) * (4 * NLOC * 4) + (u32)(c.lane & 15) * 16u;
    const u32 lt = c.lane < 16 ? ld : 240u;
    const char* csrc = nullptr; const char* cdst = nullptr; const char* covf = nullptr;
    int have = 0, flag = 0;
    for (int it = 0;; ++it) {
      LAS char* ci = img + ((it + 1) & 1) * 8192;
      const u32 cia = (u32)(uintptr_t)ci, nia = (u32)(uintptr_t)img + (u32)(it & 1) * 8192u;
#define ASK(s, i, k, l2) do { glds<((k) & 3) * 1024>((s) + ((k) >> 2) * 4096, ld, (i) + ((k) >> 2) * 4096); \
        glds<(((k) + 1) & 3) * 1024>((s) + ((k) >> 2) * 4096, l2, (i) + ((k) >> 2) * 4096); } while (0)
#define PUT(k) do { u4 a_, b_; \
        if (have == 3) { unpack_piece<true>(a_, b_, ci + (k) * PC_B, img + 16384, covf + (k) * 256, c.lane); \
          if (flag == 2) a_ = b_ = (u4){0x7fc00000u, 0x7fc00000u, 0x7fc00000u, 0x7fc00000u}; \
          st16_wt<((k) >> 1) * 256>(cdst + ((k) & 1) * (2 * NLOC * 4), lo, a_); st16_wt<((k) >> 1) * 256>(cdst + (((k) & 1) * 2 + 1) * (NLOC * 4), lo, b_); \
        } else { unpack_piece<false>(a_, b_, ci + (k) * PC_B, img + 16384, covf + (k) * 256, c.lane); \
          st16_wt<0>(cdst + (have == 2 ? ((k) & 1) * 4096 + ((k) >> 1) * 1024 : (k) * 2048), ld, a_); \
          st16_wt<0>(cdst + (have == 2 ? ((k) & 1) * 4096 + ((k) >> 1) * 1024 + 2048 : (k) * 2048 + 1024), ld, b_); } } while (0)
      if (have) { ASK(csrc, cia, 4, ld); PUT(0); wait_vm<4>(); } else wait_vm<0>();
      raw_barrier();
      const int nu = __builtin_amdgcn_readfirstlane(post[(it & 1) * 4 + (ing ? 0 : 1)]);
      flag = __builtin_amdgcn_readfirstlane(post[(it & 1) * 4 + 2]);
      if (flag == 1) break;
      if (have) { ASK(csrc, cia, 6, lt); PUT(1); }
      const int take = nu >= (ing ? UI : UO) ? 0 : (ing || (nu >= UW && nu < UX)) ? 1 : nu < UW ? 2 : 3;
      const char* nsrc; const char* ndst; const char* novf;
      if (take != 3) {
        const int kt = ing ? in_kt(nu) : (nu >> 2) & 127, w = take == 2;
        const int sq = ing ? in_qp(nu) : w ? 4 * me + (nu & 3) : 12 + (nu & 3), dq = w ? 4 * d + (nu & 3) : sq;
        const size_t un = (size_t)((w * NKT + kt) * 16 + sq);
        nsrc = pw + PK_OFF + un * (4 * PC_B); novf = pw + OVF_OFF + un * 1024;
        ndst = c.ws + c.par + (size_t)(w ? 9 : peer) * STREAM_B + XOFF(dq, kt);
      } else {
        int s_, qp_, c1_; const int t_ = unit_tile(nu - UX); remote_tile(t_, peer, me, s_, qp_, c1_);
        const size_t un = (size_t)(me * NSU + nu - UX);
        nsrc = pw + STG_OFF + un * (4 * PC_B); novf = pw + SOV_OFF + un * 1024;
        ndst = (const char*)p.out[rank] + ((size_t)(s_ * NLOC + qp_ * 64 + (nu - UX - tile_unit(t_)) * 16) * NLOC + 256 * d + 128 * c1_) * 4;
      }
      if (have) {
        wait_vm<6>();
        if (take) { ASK(nsrc, nia, 0, ld); PUT(2); wait_vm<6>(); PUT(3); ASK(nsrc, nia, 2, ld); wait_vm<6>(); }
        else { PUT(2); wait_vm<4>(); PUT(3); }
      } else if (take) { ASK(nsrc, nia, 0, ld); ASK(nsrc, nia, 2, ld); wait_vm<2>(); }
      csrc = nsrc; cdst = ndst; covf = novf; have = take;
    }
    return;
  }
  if (blockIdx.x < 256 - NLOCAL) get_call();
  else {
    const u32 ld = (u32)c.lane * 16u, lx = x_lane_off(c.lane), lw = w_lane_off(c.lane);
    const u64 peer = (u64)p.ws[(rank ^ ((c.lane & 7) + 1)) & 7];
    LAS char* scr = lds + c.wid * 4096;
    int i = blockIdx.x - (256 - NLOCAL);
    if (i >= 64) for (int k = 0; k < CVT_DELAY * (1 + (i >> 7)); ++k) __builtin_amdgcn_s_sleep(32);
#define CL(u) do { const bool w_ = ((u) & 8) != 0; \
      const char* s_ = w_ ? w_src(p.w, 2 * ((u) & 7) + (c.wid & 1), 2 * ((u) >> 4) + ((c.wid >> 1) & 1)) + (size_t)(c.wid >> 2) * 32 * NLOC * 4 \
                          : x_src(p.x, 8 * ((u) & 7) + c.wid, 2 * ((u) >> 4)); \
      const u32 l_ = w_ ? lw : lx, t_ = w_ ? NLOC * 4 : 64; const char* z_ = s_ + (w_ ? 16 * NLOC * 4 : 256); \
      ld16<0>(r[0], s_, l_); ld16<0>(r[1], s_, l_ + t_); ld16<0>(r[2], s_, l_ + 2 * t_); ld16<0>(r[3], s_, l_ + 3 * t_); \
      ld16<0>(r[4], z_, l_); ld16<0>(r[5], z_, l_ + t_); ld16<0>(r[6], z_, l_ + 2 * t_); ld16<0>(r[7], z_, l_ + 3 * t_); } while (0)
    for (bool first = true; i < 1024; i += NLOCAL, first = false) {
      const int half = i & 7, w = (i >> 3) & 1;
      f4 r[8];
      CL(i);
      if (first) get_call();
      wait_vm<0>(); pin4(r); pin4(r + 4);
      if (i < 2 * NLOCAL) STAMP(40 + 4 * (i >= NLOCAL));
      u4 cv[4];
      if (w) {
#pragma unroll
        for (int j = 0; j < 4; ++j) cv[j] = cvt8(r[0][j], r[1][j], r[2][j], r[3][j], r[4][j], r[5][j], r[6][j], r[7][j]);
      } else {
#pragma unroll
        for (int j = 0; j < 4; ++j) cv[j] = cvt8(r[2 * j], r[2 * j + 1]);
      }
      const int kt = 2 * (i >> 4) + (w ? (c.wid >> 1) & 1 : 0), blk = w ? 4 * (2 * half + (c.wid & 1)) : 8 * half + c.wid, kk = w ? c.wid >> 2 : 0;
      const char* dd = c.ws + c.par + (size_t)(w ? 8 : rank) * STREAM_B + (size_t)kt * KT_B + blk * 2048 + kk * 1024;
      st16_wt<0>(dd, ld, cv[0]); st16_wt<0>(dd + (w ? 2048 : 1024), ld, cv[1]);
      st16_wt<0>(dd + (w ? 4096 : KT_B), ld, cv[2]); st16_wt<0>(dd + (w ? 6144 : KT_B + 1024), ld, cv[3]);
      {
        const int pc = (w * NKT + kt) * 64 + blk + 2 * kk, ps = w ? 1 : 64;
        pack_piece<false>(cv[0], cv[1], scr, c.ws + c.par + PK_OFF + (size_t)pc * PC_B, c.ws + c.par + OVF_OFF + (size_t)pc * 256, c.lane);
        pack_piece<false>(cv[2], cv[3], scr, c.ws + c.par + PK_OFF + (size_t)(pc + ps) * PC_B, c.ws + c.par + OVF_OFF + (size_t)(pc + ps) * 256, c.lane);
      }
      wait_vm<0>(); raw_barrier();
      if (i < 2 * NLOCAL) STAMP(41 + 4 * (i >= NLOCAL));
      if (c.wid == 0) {
        if (c.lane < 4) st_sys((GAS u32*)(c.ws + FLG_OFF + (w ? 8 : rank) * FLG_B) + (2 * half + (c.lane >> 1)) * NKT + 2 * (i >> 4) + (c.lane & 1), c.call);
        u32 n = 0;
        if (c.lane == 0) n = __hip_atomic_fetch_add((GAS u32*)(c.ws + CVC_OFF + (i >> 5) * 128), 1u, __ATOMIC_RELAXED, __HIP_MEMORY_SCOPE_AGENT);
        if ((u32)__builtin_amdgcn_readfirstlane(n) == 32u * c.call - 1u && c.lane < 7) st_sys((GAS u32*)(peer + CVT_OFF + (rank * 32 + (i >> 5)) * 4), c.call);
        if (i < 2 * NLOCAL) STAMP(42 + 4 * (i >= NLOCAL));
      }
    }
    raw_barrier();
  }
  STAMP(1);
  const int li = slot - NCS;
  if (li >= NGS) return;
  const int hp = xcd & 1, q = xcd >> 1;
  auto make_task = [&](int src, int qp, int c1, int wstream) {
    const int ch = 2 * q + c1;
    Task t;
    t.a = c.ws + c.par + (size_t)src * STREAM_B + qp * 8192;
    t.b = c.ws + c.par + (size_t)wstream * STREAM_B + ch * 16384;
    t.fa = c.ws + FLG_OFF + src * FLG_B + qp * 512;
    t.fb = c.ws + FLG_OFF + wstream * FLG_B + ch * 1024;
    t.out = (const char*)(p.out[rank] + ((size_t)src * NLOC + qp * 64) * NLOC + 128 * ch); t.ovf = nullptr; t.done = nullptr;
    return t;
  };
  auto stage = [&](Task& t, int rt) { const size_t u = q * NSU + tile_unit(rt);
    t.out = c.ws + c.par + STG_OFF + u * (4 * PC_B); t.ovf = c.ws + c.par + SOV_OFF + u * 1024;
    t.done = p.ws[og + q] + OST_OFF + (rt * 4 + me) * 4; };
  for (int ph = 0; ph < 2; ++ph) {
    int src, qp, c1, rt = -1;
    if (ph == 0) {
      if (li < 14) { const int r = li >> 1; c1 = li & 1; if (r < 4) { src = rank; qp = 4 * r + 2 * hp; } else { src = rank ^ (r - 3); qp = 2 * hp; } }
      else rt = 2 * (li - 14) + hp;
    } else if (li < 12) {
      const int a = li >> 2; c1 = li & 1; src = rank ^ (a + 1); qp = 4 + 2 * hp;
      if (li & 2) rt = 2 * (12 + a * 2 + c1) + hp;
    } else {
      const int j = li - 12;
      if (j < 8) { src = og + (j >> 1); c1 = j & 1; qp = 12 + 2 * hp; }
      else rt = 2 * (18 + j - 8) + hp;
    }
    if (rt >= 0) remote_tile(rt, rank, q, src, qp, c1);
    Task t = make_task(src & 7, qp, c1, rt >= 0 ? 9 : 8);
    if (rt >= 0) stage(t, rt);
    gemm_task<4>(c, lds, t, 8 + 8 * ph);
    STAMP(2 + ph);
  }
  if (li < 24) {
    const int last = li >= 12, j = li - 12 * last, a = j >> 2, c1 = j & 1, qp = 8 + 2 * last + hp, src = (rank ^ (a + 1)) & 7;
    const bool rem = (j & 2) != 0;
    Task t = make_task(src, qp, c1, rem ? 9 : 8);
    if (rem && !last) stage(t, 48 + a * 4 + c1 * 2 + hp);
    if (rem && last) {
      t.out = (const char*)(p.out[og + q] + ((size_t)src * NLOC + qp * 64) * NLOC + 256 * me + 128 * c1);
      t.done = p.ws[og + q] + DON_OFF + (me * 16 + a * 4 + c1 * 2 + hp) * 4;
    }
    gemm_task<2>(c, lds, t, 24);
    STAMP(4);
  }
}

}

extern "C" void kernel_launch(int rank, int nranks, ncclComm_t comm,
                              hipStream_t stream,
                              void* const* d_in, const int* in_sizes, int n_in,
                              void* d_out, int out_size,
                              const int* in_dtypes, int out_dtype,
                              void* const* d_out_all,
                              void* const* d_ws_all, size_t ws_bytes) {
  if (nranks != NR || n_in != 2 || ws_bytes < WS_NEED || (unsigned)rank >= (unsigned)NR ||
      in_sizes[0] != NLOC * KDIM || in_sizes[1] != KDIM * NLOC || out_size != KDIM * NLOC) {
    fprintf(stderr, "unexpected problem shape\n");
    return;
  }
  static bool attr = false;
  if (!attr) {
    HIP_CHECK(hipFuncSetAttribute((const void*)ag_gemm, hipFuncAttributeMaxDynamicSharedMemorySize, LDS_B));
    attr = true;
  }
  Params p;
  p.x = (const float*)d_in[0]; p.w = (const float*)d_in[1];
  for (int r = 0; r < NR; ++r) { p.ws[r] = (char*)d_ws_all[r]; p.out[r] = (float*)d_out_all[r]; }
  p.rank = rank; p.pad = 0;
  hipLaunchKernelGGL(ag_gemm, dim3(256), dim3(512), LDS_B, stream, p);
  HIP_CHECK(hipGetLastError());
}
```

```cpp
#include <hip/hip_runtime.h>
#include <rccl/rccl.h>
#include <cstdio>
#include <cstdint>

#define HIP_CHECK(expr) do { hipError_t _e = (expr); if (_e != hipSuccess) { \
    fprintf(stderr, "HIP error %s at %s:%d\n", hipGetErrorString(_e), __FILE__, __LINE__); \
    return; } } while (0)

typedef unsigned u32;
typedef unsigned long long u64;
typedef _Float16 h8 __attribute__((ext_vector_type(8)));
typedef float f4 __attribute__((ext_vector_type(4)));
typedef unsigned u4 __attribute__((ext_vector_type(4)));
typedef unsigned u2 __attribute__((ext_vector_type(2)));
#define GAS __attribute__((address_space(1)))
#define LAS __attribute__((address_space(3)))

#define NC_SLOTS 5
#define NB4 4
#define NB2 4
#define NBQ 3
#define LAST_SLEEP 8
#define NP 24
#define PWW 3
#define PWX 2
#define PA 2
#define PB 1
#define CVT_DELAY 2
#define POLL_SLEEP 8

namespace {

constexpr int NR = 8;
constexpr int KDIM = 8192, NLOC = 1024;
constexpr int NKT = KDIM / 64;
constexpr int NC = 8 * NC_SLOTS, NCS = NC_SLOTS;
constexpr int NGS = 27;
static_assert(NCS + NGS <= 32 && NC <= 64 && NC >= 36, "grid");
constexpr int NLOCAL = 256 - NC;
constexpr int NST = 72;
constexpr int NSS = 50 * 8 + 22 * 4;
constexpr int UI = 1472;
constexpr int UW = 512, UX = 1088, UO = UX + NSS - 12 - 36;
constexpr size_t CTR_OFF = 0;
constexpr size_t TMO_OFF = 32768;
constexpr size_t CLM_OFF = 32768 + 256;
constexpr size_t TSK_OFF = 40960;
constexpr size_t CVT_OFF = 57344;
constexpr size_t CVC_OFF = 45056;
constexpr size_t EFL_OFF = 59392;
constexpr size_t OST_OFF = 36864;
constexpr size_t FLG_OFF = 65536;
constexpr size_t FLG_B = 8192;
constexpr size_t DBG_OFF = 262144;
#define DATA_MB 1
constexpr size_t DATA_OFF = (size_t)DATA_MB << 20;
#define SKEW 36864
constexpr size_t KT_B = 32768;
constexpr size_t PANEL_B = (size_t)NKT * KT_B + SKEW;
constexpr size_t STREAM_B = 4 * PANEL_B;
constexpr int NSTREAM = 10;
constexpr int PC_B = 1664, PF_B = 1760;
#define UNIT_B 7680
constexpr int SL_B = UNIT_B;
constexpr size_t KTP_B = 16 * SL_B;
constexpr size_t PSTREAM_B = (size_t)NKT * KTP_B;
constexpr size_t PK_OFF = 0;
constexpr size_t OVF_OFF = PK_OFF + 2 * PSTREAM_B;
constexpr size_t STG_OFF = OVF_OFF + (size_t)2 * NKT * 64 * 2048;
constexpr size_t SOV_OFF = STG_OFF + (size_t)4 * NSS * SL_B;
constexpr size_t PAR_B = SOV_OFF + (size_t)4 * NSS * 4 * 2048;
constexpr size_t PUB_OFF = DATA_OFF + (NSTREAM + 2) * STREAM_B;
constexpr size_t WS_NEED = PUB_OFF + 2 * PAR_B;
constexpr int LDS_B = NB4 * 32768 > NB2 * 24576 ? NB4 * 32768 : NB2 * 24576;
constexpr u64 DEADLINE = 50000000ull;

struct Params {
  const float* x; const float* w;
  char* ws[NR]; float* out[NR];
  int rank; int pad;
};

#define FENCE() asm volatile("" ::: "memory")
#define SB() __builtin_amdgcn_sched_barrier(0)
__device__ __forceinline__ void raw_barrier() { FENCE(); __builtin_amdgcn_s_barrier(); FENCE(); }
template <int N> __device__ __forceinline__ void wait_vm() { asm volatile("s_waitcnt vmcnt(%0)" :: "n"(N) : "memory"); }
__device__ __forceinline__ void wait_lgkm() { FENCE(); __builtin_amdgcn_s_waitcnt(0xC07F); FENCE(); }
constexpr int waitcnt(int vm, int lgkm) { return (vm & 15) | (7 << 4) | ((lgkm & 15) << 8) | ((vm >> 4) << 14); }

__device__ __forceinline__ u32 ld_sys(const GAS u32* p) { return __hip_atomic_load(p, __ATOMIC_RELAXED, __HIP_MEMORY_SCOPE_SYSTEM); }
__device__ __forceinline__ u32 ld_agent(const GAS u32* p) { return __hip_atomic_load(p, __ATOMIC_RELAXED, __HIP_MEMORY_SCOPE_AGENT); }
__device__ __forceinline__ void st_sys(GAS u32* p, u32 v) { __hip_atomic_store(p, v, __ATOMIC_RELAXED, __HIP_MEMORY_SCOPE_SYSTEM); }
__device__ __forceinline__ void st_agent(GAS u32* p, u32 v) { __hip_atomic_store(p, v, __ATOMIC_RELAXED, __HIP_MEMORY_SCOPE_AGENT); }

template <int OFF> __device__ __forceinline__ void st16_wt(const char* base, u32 voff, u4 v) {
  asm volatile("global_store_dwordx4 %0, %1, %2 offset:%3 sc0 sc1\n\ts_nop 1" :: "v"(voff), "v"(v), "s"(base), "n"(OFF) : "memory");
}
#define IN_BITS " nt"
#define OUT_BITS " sc0 sc1 nt"
template <int OFF> __device__ __forceinline__ void st16_out(const char* base, u32 voff, u4 v) {
  asm volatile("global_store_dwordx4 %0, %1, %2 offset:%3" OUT_BITS "\n\ts_nop 1" :: "v"(voff), "v"(v), "s"(base), "n"(OFF) : "memory");
}
template <int OFF> __device__ __forceinline__ void ld16(f4& d, const char* base, u32 voff) {
  asm volatile("global_load_dwordx4 %0, %1, %2 offset:%3" IN_BITS : "=v"(d) : "v"(voff), "s"(base), "n"(OFF) : "memory");
}
__device__ __forceinline__ u4 cvt8(float a0, float a1, float a2, float a3, float a4, float a5, float a6, float a7) {
  h8 r; r[0] = (_Float16)a0; r[1] = (_Float16)a1; r[2] = (_Float16)a2; r[3] = (_Float16)a3;
  r[4] = (_Float16)a4; r[5] = (_Float16)a5; r[6] = (_Float16)a6; r[7] = (_Float16)a7;
  return __builtin_bit_cast(u4, r);
}
__device__ __forceinline__ u4 cvt8(const f4& a, const f4& b) { return cvt8(a[0], a[1], a[2], a[3], b[0], b[1], b[2], b[3]); }
__device__ __forceinline__ void pin4(f4* r) { asm volatile("" : "+v"(r[0]), "+v"(r[1]), "+v"(r[2]), "+v"(r[3]) :: "memory"); }

#define DBG_PUT(i, val) do { } while (0)
#define STAMP(i) DBG_PUT(i, __builtin_amdgcn_s_memrealtime())

struct Ctx {
  char* ws;
  GAS u32* tmo;
  u32 call;
  int wid, lane;
  size_t par;
};

__device__ __forceinline__ bool give_up(const Ctx& c, u64 t0) {
  if (__builtin_amdgcn_readfirstlane(ld_agent(c.tmo)) == c.call || __builtin_amdgcn_s_memrealtime() - t0 > DEADLINE) {
    st_agent(c.tmo, c.call); return true;
  }
  return false;
}

__device__ __forceinline__ u32 x_lane_off(int lane) { return (u32)((lane >> 2) * KDIM + 4 * ((lane & 3) ^ ((-(lane >> 4)) & 3))) * 4u; }
__device__ __forceinline__ const char* x_src(const float* x, int rb, int kt) { return (const char*)x + ((size_t)rb * 16 * KDIM + (size_t)kt * 64) * 4; }
__device__ __forceinline__ void x_load(f4* r, const char* s, u32 lo) { ld16<0>(r[0], s, lo); ld16<64>(r[1], s, lo); ld16<128>(r[2], s, lo); ld16<192>(r[3], s, lo); }
__device__ __forceinline__ u32 w_lane_off(int lane) { return (u32)(4 * (lane >> 4) * NLOC + 4 * (lane & 15)) * 4u; }
__device__ __forceinline__ const char* w_src(const float* w, int g, int kt) { return (const char*)w + ((size_t)kt * 64 * NLOC + (size_t)g * 64) * 4; }

__device__ __forceinline__ void w_load(f4* r, const char* s, u32 l0) {
  const u32 l1 = l0 + NLOC * 4, l2 = l0 + 2 * NLOC * 4, l3 = l0 + 3 * NLOC * 4;
#pragma unroll
  for (int i = 0; i < 4; ++i) { const char* t = s + (size_t)i * 16 * NLOC * 4;
    ld16<0>(r[4 * i], t, l0); ld16<0>(r[4 * i + 1], t, l1); ld16<0>(r[4 * i + 2], t, l2); ld16<0>(r[4 * i + 3], t, l3); }
}
__device__ __forceinline__ void w_cvt(u4* cv, const f4* r) {
#pragma unroll
  for (int j = 0; j < 4; ++j)
#pragma unroll
    for (int kk = 0; kk < 2; ++kk)
      cv[2 * j + kk] = cvt8(r[8 * kk][j], r[8 * kk + 1][j], r[8 * kk + 2][j], r[8 * kk + 3][j], r[8 * kk + 4][j], r[8 * kk + 5][j], r[8 * kk + 6][j], r[8 * kk + 7][j]);
}

constexpr u32 B1 = 0x01010101u;
__device__ __forceinline__ u32 bfi(u32 m, u32 a, u32 b) { return (a & m) | (b & ~m); }
__device__ __forceinline__ u32 perm(u32 hi, u32 lo, u32 sel) { return __builtin_amdgcn_perm(hi, lo, sel); }
template <int CTRL, int RM> __device__ __forceinline__ u32 dpp(u32 v) { return (u32)__builtin_amdgcn_update_dpp(0, (int)v, CTRL, RM, 0xf, true); }
__device__ __forceinline__ u32 wave_scan(u32 v) {
  v += dpp<0x111, 0xf>(v); v += dpp<0x112, 0xf>(v); v += dpp<0x114, 0xf>(v); v += dpp<0x118, 0xf>(v);
  v += dpp<0x142, 0xa>(v); v += dpp<0x143, 0xc>(v);
  return v;
}
__device__ __forceinline__ u32 wave_max(u32 v) {
  v = max(v, dpp<0x111, 0xf>(v)); v = max(v, dpp<0x112, 0xf>(v)); v = max(v, dpp<0x114, 0xf>(v)); v = max(v, dpp<0x118, 0xf>(v));
  v = max(v, dpp<0x142, 0xa>(v)); v = max(v, dpp<0x143, 0xc>(v));
  return (u32)__builtin_amdgcn_readlane(v, 63);
}
__device__ __forceinline__ u32 opaque(u32 v) { asm("" : "+v"(v)); return v; }
__device__ __forceinline__ u32 dot4(u32 v, u32 w, u32 a) { return __builtin_amdgcn_udot4(v, w, a, false); }
__device__ __forceinline__ u32 gather2(u32 v) { return dot4(v, 0x40100401u, 0u); }
__device__ __forceinline__ u32 gather1(u32 v) { return dot4(v, 0x08040201u, 0u); }
__device__ __forceinline__ u32 fill(u32 f) { return opaque(f << 8) - f; }
__device__ __forceinline__ u32 spread2(u32 tab, u32 v, u32 at) { return *(const LAS u32*)(uintptr_t)(tab + 4u * __builtin_amdgcn_ubfe(v, at, 8)); }
__device__ __forceinline__ u32 placer(u32 tab, u32 f, u32 w) { return *(const LAS u32*)(uintptr_t)(dot4(f, w, tab) + 1024u); }
__device__ __forceinline__ u32 lds_bits(const LAS char* im, u32 bit) {
  const LAS u32* w = (const LAS u32*)(im + ((bit >> 3) & ~3u));
  return __builtin_amdgcn_alignbit(w[1], w[0], bit & 31u);
}
__device__ __forceinline__ void lds_or_bits(LAS char* im, u32 bit, u32 v) {
  const u64 x = (u64)v << (bit & 31u);
  LAS u32* w = (LAS u32*)(im + ((bit >> 3) & ~3u));
  __hip_atomic_fetch_or(w, (u32)x, __ATOMIC_RELAXED, __HIP_MEMORY_SCOPE_WORKGROUP);
  __hip_atomic_fetch_or(w + 1, (u32)(x >> 32), __ATOMIC_RELAXED, __HIP_MEMORY_SCOPE_WORKGROUP);
}
__device__ __forceinline__ void load_plain(u4& a, u4& b, const char* ovf, int lane) {
  asm volatile("global_load_dwordx4 %0, %2, %3 sc0 sc1\n\tglobal_load_dwordx4 %1, %2, %3 offset:1024 sc0 sc1\n\ts_waitcnt vmcnt(0)"
               : "=&v"(a), "=&v"(b) : "v"((u32)lane * 16u), "s"(ovf) : "memory");
}
struct Coded { u32 C[4], w2, w3, pos, hdr; int n2, n3; };
__device__ __forceinline__ Coded code16(u4 a, u4 b, const LAS u32* lut) {
  const u32 H[4] = {perm(a[1], a[0], 0x07050301u), perm(a[3], a[2], 0x07050301u), perm(b[1], b[0], 0x07050301u), perm(b[3], b[2], 0x07050301u)};
  const u32 sum = (u32)__builtin_amdgcn_readlane(wave_scan(__builtin_amdgcn_sad_u8(H[0] & (0x7c * B1), 0u, 0u)), 63);
  const u32 lo = (u32)__builtin_amdgcn_readfirstlane(min(sum >> 10, 28u)), mode = (sum & 1023u) < 410u;
  const u32 tlo = mode ? 0x1c1d1e3fu : 0x1c1d3f3fu, thi = mode ? 0x3f080400u : 0x1e080400u;
  Coded o; o.w2 = o.w3 = 0;
  u32 c2 = 0, c3 = 0, bad = 0, d[4], t[4], s2[4], s3[4];
#pragma unroll
  for (int j = 0; j < 4; ++j) {
    d[j] = ((H[j] >> 2) & (0x1f * B1)) + (68u - lo) * B1;
    const u32 out = (((d[j] ^ (0x40 * B1)) & (0x78 * B1)) + 0x78 * B1) & (0x80 * B1);
    t[j] = perm(thi, tlo, d[j] & (7 * B1)) | (out - (out >> 7));
    s2[j] = lut[gather1(t[j] & (0x10 * B1)) >> 4]; s3[j] = lut[gather1(t[j] & (0x20 * B1)) >> 5];
  }
  SB();
#pragma unroll
  for (int j = 0; j < 4; ++j) {
    o.C[j] = bfi(0x7c * B1, t[j] & (0x0c * B1), H[j]);
    const u32 up = (d[j] + 60 * B1) & (0x80 * B1);
    d[j] = 132 * B1 - d[j] + (up >> 5) + (up >> 7);
    bad |= (((d[j] ^ (0x40 * B1)) & (0x70 * B1)) + 0x70 * B1) & (t[j] << 2) & (0x80 * B1);
  }
  SB();
#pragma unroll
  for (int j = 0; j < 4; ++j) {
    o.w2 |= gather2(perm(0u, t[j] & (3 * B1), s2[j])) << (2 * c2);
    const u32 x = perm(0u, d[j] & (15 * B1), s3[j]);
    o.w3 |= (dot4(x, 0x00001001u, 0u) | (dot4(x, 0x10010000u, 0u) << 8)) << (4 * (c3 & 7u));
    c2 += __builtin_popcount(t[j] & (0x10 * B1)); c3 += __builtin_popcount(t[j] & (0x20 * B1));
  }
  const u32 cnt = c2 | (c3 << 16), inc = wave_scan(cnt), tot = (u32)__builtin_amdgcn_readlane(inc, 63);
  o.pos = inc - cnt; o.hdr = lo | (mode << 5);
  o.n2 = __any(bad != 0 || c3 > 8u) ? -1 : (int)(((tot & 0xffffu) + 3u) >> 2); o.n3 = (int)(((tot >> 16) + 1u) >> 1);
  return o;
}
__device__ __forceinline__ void place16(const Coded& o, u4 a, u4 b, int k, u64 sz, LAS char* im, const char* ovf, int lane) {
  int top = 508; bool fits = false;
  for (int i = 0; i <= k; ++i) {
    const int n = (int)(sz >> (16 * i)) & 0xffff;
    fits = top - n >= (i < 3 ? 128 : 0);
    if (fits) top -= n;
  }
  if (lane == 0) __hip_atomic_fetch_or((LAS u32*)(im + 508), ((fits ? o.hdr : 0x80u) << (8 * k)) | (top < 128 ? 64u : 0u), __ATOMIC_RELAXED, __HIP_MEMORY_SCOPE_WORKGROUP);
  if (!fits) { st16_wt<0>(ovf, (u32)lane * 16u, a); st16_wt<1024>(ovf, (u32)lane * 16u, b); return; }
  lds_or_bits(im, 8u * (top + o.n3) + 2u * (o.pos & 0xffffu), o.w2);
  lds_or_bits(im, 8u * top + 4u * (o.pos >> 16), o.w3);
  LAS char* pl = im + 512 + k * PC_B;
  *(LAS u4*)(pl + lane * 16) = (u4){perm(a[1], a[0], 0x06040200u), perm(a[3], a[2], 0x06040200u), perm(b[1], b[0], 0x06040200u), perm(b[3], b[2], 0x06040200u)};
  *(LAS u2*)(pl + 1024 + lane * 8) = (u2){o.C[0] | ((o.C[2] >> 1) & (0x40 * B1)) | ((o.C[2] << 2) & (0x30 * B1)), o.C[1] | ((o.C[3] >> 1) & (0x40 * B1)) | ((o.C[3] << 2) & (0x30 * B1))};
  *(LAS unsigned short*)(pl + 1536 + lane * 2) = (unsigned short)(gather2(o.C[2] & (3 * B1)) | (gather2(o.C[3] & (3 * B1)) << 8));
}
#define ALL(stmt) _Pragma("unroll") for (int j = 0; j < 4; ++j) { stmt; }
template <class F>
__device__ __forceinline__ void unpack16(u4& a, u4& b, int k, const LAS char* im, u32 tab, int& top, u32 hdr, const char* ovf, int lane, F mid) {
  const u32 h = hdr >> (8 * k), lo = h & 31u;
  mid();
  if (h & 0x80u) { load_plain(a, b, ovf, lane); return; }
  const LAS char* pl = im + 512 + k * PC_B;
  const u2 r = *(const LAS u2*)(pl + 1024 + lane * 8);
  const u32 m2 = *(const LAS unsigned short*)(pl + 1536 + lane * 2);
  const u4 q = *(const LAS u4*)(pl + lane * 16);
  const u32 flip = *(const LAS u32*)(uintptr_t)(tab + 1280u + 4u * (h & 63u));
  u32 H[4], f[4], c[5], x[4], y[4];
  ALL(H[j] = (j < 2 ? r[j] : r[j - 2] >> 2) & (0x0c * B1));
  ALL(f[j] = (H[j] + 4 * B1) & (0x10 * B1));
  c[0] = 0;
  ALL(c[j + 1] = c[j] + __builtin_popcount(f[j]));
  ALL(y[j] = placer(tab, f[j], 0x08040201u));
  const u32 s0 = spread2(tab, m2, 0), s1 = spread2(tab, m2, 8);
  u32 inc = wave_scan(c[4]);
  top -= (int)(((u32)__builtin_amdgcn_readlane(inc, 63) + 3u) >> 2);
  u32 w = lds_bits(im, 8u * top + 2u * (inc - c[4]));
  ALL(H[j] += lo * (4 * B1));
  H[0] = bfi(0x7c * B1, H[0], r[0]); H[1] = bfi(0x7c * B1, H[1], r[1]);
  H[2] |= ((r[0] << 1) & (0x80 * B1)) | s0; H[3] |= ((r[1] << 1) & (0x80 * B1)) | s1;
  ALL(x[j] = spread2(tab, w, 2 * c[j]));
  ALL(x[j] = perm(0u, perm(0u, flip, x[j]), y[j]));
  ALL(f[j] = x[j] & B1; H[j] ^= x[j] & (0x7c * B1));
  ALL(c[j + 1] = c[j] + __builtin_popcount(f[j]));
  ALL(y[j] = placer(tab, f[j], 0x80402010u));
  inc = wave_scan(c[4]);
  top -= (int)(((u32)__builtin_amdgcn_readlane(inc, 63) + 1u) >> 1);
  w = lds_bits(im, 8u * top + 4u * (inc - c[4]));
  auto exps = [&](u32 sel) {
    const u32 n = perm(w >> 2, w << 2, sel) & (0x3c * B1), far = (n + 0x74 * B1) & (0x80 * B1);
    return ((4u * lo + 148u) * B1 - n - (far >> 3) - (far >> 5)) & (0x7c * B1);
  };
  u64 X = exps(0x05010400u);
  if (__any(c[4] > 4u)) X |= (u64)exps(0x07030602u) << 32;
  ALL(H[j] ^= perm(0u, (u32)(X >> (8 * c[j])), y[j]));
  a = (u4){perm(H[0], q[0], 0x05010400u), perm(H[0], q[0], 0x07030602u), perm(H[1], q[1], 0x05010400u), perm(H[1], q[1], 0x07030602u)};
  b = (u4){perm(H[2], q[2], 0x05010400u), perm(H[2], q[2], 0x07030602u), perm(H[3], q[3], 0x05010400u), perm(H[3], q[3], 0x07030602u)};
}
__device__ __forceinline__ void pack32(u4 a, u4 b, LAS char* scr, const char* dst, const char* ovf, int lane) {
  const u32 H[2] = {perm(perm(a[3], a[2], 0x07030703u), perm(a[1], a[0], 0x07030703u), 0x05040100u),
                    perm(perm(b[3], b[2], 0x07030703u), perm(b[1], b[0], 0x07030703u), 0x05040100u)};
  u32 mn = 0, mp = 0;
#pragma unroll
  for (int k = 0; k < 8; ++k) { const u32 x = (H[k & 1] >> (8 * (k >> 1))) & 0xffu; mn = max(mn, x); mp = max(mp, x ^ 0x80u); }
  const int tp = (int)wave_max(mp) - 131, tn = (int)wave_max(mn) - 130;
  const u32 bp = (u32)__builtin_amdgcn_readfirstlane(tp < 0 ? 0 : tp), bn = (u32)__builtin_amdgcn_readfirstlane(tn < 0 ? 0 : tn);
  u32 m = 0, c16 = 0, c8 = 0;
#pragma unroll
  for (int j = 0; j < 2; ++j) {
    const u32 s = (H[j] >> 7) & B1;
    const u32 u = (H[j] | (0x80 * B1)) - bfi(fill(s), bn * B1, bp * B1);
    const u32 e = (~u >> 7) & B1;
    c16 |= gather2((u | e | opaque(e << 1)) & (3 * B1)) << (8 * j);
    c8 |= gather2(s | e) << j;
    m |= gather1(e) << (4 * j);
  }
  if (lane == 0) *(LAS u32*)(scr + 704) = bp | (bn << 8);
  const u32 cnt = __builtin_popcount(m), inc = wave_scan(cnt);
  if ((u32)__builtin_amdgcn_readlane(inc, 63) > 30u) {
    st16_wt<0>(ovf, (u32)lane * 16u, a); st16_wt<1024>(ovf, (u32)lane * 16u, b);
    if (lane == 0) *(LAS u32*)(scr + 704) = 255u;
  } else {
    for (u32 pos = inc - cnt + 706u; m; ++pos, m &= m - 1) scr[pos] = (char)perm(H[1], H[0], __builtin_ctz(m));
  }
  st16_wt<0>(dst, (u32)lane * 16u, (u4){perm(b[2], a[0], 0x04020100u), perm(b[2], a[1], 0x05020100u), perm(b[2], a[2], 0x06020100u), perm(b[3], a[3], 0x04020100u)});
  *(LAS u2*)(scr + lane * 8) = (u2){perm(b[3], b[0], 0x05020100u), perm(b[3], b[1], 0x06020100u)};
  *(LAS unsigned short*)(scr + 512 + lane * 2) = (unsigned short)c16; scr[640 + lane] = (char)c8;
  if (lane < 46) st16_wt<1024>(dst, (u32)lane * 16u, *(const LAS u4*)(scr + lane * 16));
}
template <class F>
__device__ __forceinline__ void unpack32(u4& a, u4& b, const LAS char* pl, u32 tab, const char* ovf, int lane, F mid) {
  const u4 q = *(const LAS u4*)(pl + lane * 16); const u2 r = *(const LAS u2*)(pl + 1024 + lane * 8);
  const u32 c16 = *(const LAS unsigned short*)(pl + 1536 + lane * 2), c8 = *(const LAS unsigned char*)(pl + 1664 + lane);
  const u32 h = (u32)__builtin_amdgcn_readfirstlane(*(const LAS u32*)(pl + 1728)), bp = h & 0xffu, bn = (h >> 8) & 0xffu;
  mid();
  if (bp == 255u) { load_plain(a, b, ovf, lane); return; }
  u32 H[2], f[2], y[2];
  const u32 t = spread2(tab, c8, 0);
#pragma unroll
  for (int j = 0; j < 2; ++j) {
    const u32 l = spread2(tab, c16, 8 * j), s = (t >> j) & B1;
    H[j] = l + bfi(fill(s), (bn + 128u) * B1, bp * B1);
    f[j] = ((l + B1) >> 2) & s;
    y[j] = placer(tab, f[j], 0x80402010u);
  }
  const u32 c0 = __builtin_popcount(f[0]), cnt = c0 + __builtin_popcount(f[1]), bit = 8u * (wave_scan(cnt) - cnt + 1730u);
  const u64 X = lds_bits(pl, bit) | (u64)lds_bits(pl, bit + 32u) << 32;
  H[0] = bfi(fill(f[0]), perm(0u, (u32)X, y[0]), H[0]);
  H[1] = bfi(fill(f[1]), perm(0u, (u32)(X >> (8 * c0)), y[1]), H[1]);
  a = (u4){perm(H[0], q[0], 0x04020100u), perm(H[0], q[1], 0x05020100u), perm(H[0], q[2], 0x06020100u), perm(H[0], q[3], 0x07020100u)};
  b = (u4){perm(H[1], r[0], 0x04020100u), perm(H[1], r[1], 0x05020100u),
           perm(perm(H[1], q[2], 0x06030603u), perm(q[1], q[0], 0x07030703u), 0x05040100u),
           perm(perm(H[1], r[1], 0x07030703u), perm(r[0], q[3], 0x07030703u), 0x05040100u)};
}

struct Unit { int kind, grp, kt; };
__device__ __forceinline__ Unit in_unit(int u) {
  if (u < 1024) return {1, 4 * (u >> 9) + (u & 3), (u >> 2) & 127};
  constexpr int PER = 4 * PA + 3 * PB, HEAD = PER * (64 / PA);
  const int v = u - 1024;
  int b;
  if (v < HEAD) {
    const int p = v / PER, w = v % PER;
    if (w < 4 * PA) return {1, 8 + (w & 1), 2 * PA * p + (w >> 1)};
    b = 3 * PB * p + w - 4 * PA;
  } else b = 3 * PB * (64 / PA) + v - HEAD;
  const int idx = b / 3, r = b % 3;
  return {r < 2 ? 1 : 4, r < 2 ? 10 : 11, (2 * idx + (r & 1)) & 127};
}
__device__ __forceinline__ Unit out_unit(int u) {
  if (u < UW) return {2, u & 3, u >> 2};
  if (u >= UX) return {3, 0, 0};
  const int p = (u - UW) / 9, r = (u - UW) % 9;
  return {r < 8 ? 1 : 4, r < 8 ? 12 + (r & 3) : 11, 2 * p + ((r >> 2) & 1)};
}

__device__ __forceinline__ u32 rows128(int src, int first) { return (u32)((src & 7) * 32 + first) * 0x01010101u + 0x03020100u; }
__device__ __forceinline__ u32 last_rows(int k, int r) {
  return k < 3 ? rows128(r ^ (k + 1), 20) : (u32)(((r ^ (2 * k - 5)) & 7) * 32 + 22) | (u32)(((r ^ (k - 1)) & 7) * 32 + 22) << 8;
}
__device__ __forceinline__ u32 tile_rows(int t, int r, int q) {
  const int og = (r & 4) ^ 4, hp = (t >> 1) & 1, j = t >> 2;
  if (t < 12) return rows128(r ^ (j + 1), 4 * hp);
  if (t < 24) return rows128(r, 8 * (j - 3) + 4 * hp);
  if (t < 36) return rows128(r ^ (j - 5), 8 + 4 * hp);
  if (t < 48) return rows128(og + (j - 9) + (j - 9 >= q ? 1 : 0), 24 + 4 * hp);
  if (t < 50) {
    const u32 a = (og + (q == 0 ? 1 : 0)) * 32 + 23, b = (og + (q <= 1 ? 2 : 1)) * 32 + 23, e = (og + (q <= 2 ? 3 : 2)) * 32 + 23;
    return a | (b << 8) | (e << 16) | (e << 24);
  }
  if (t < 62) return rows128(r ^ (((t - 50) >> 2) + 1), 16 + ((t - 50) & 2));
  return last_rows((t - 62) >> 1, r);
}
__device__ __forceinline__ int out_slot(int u) { const int v = u - UX; return v + (v < 182 ? 0 : v < 188 ? 2 : v < 386 ? 4 : v < 392 ? 6 : v < 474 ? 8 : 10); }
__device__ __forceinline__ int slot_tile(int s) { return s < 400 ? s >> 3 : 50 + ((s - 400) >> 2); }

__device__ __forceinline__ size_t x_panel(const Ctx& c, int s, int panel) {
  return DATA_OFF + (size_t)(panel == 0 && (c.call & 1u) ? 4 * NSTREAM + s : 4 * s + panel) * PANEL_B;
}
struct Task {
  u32 rows;
  int nruns;
  int wstream, cg;
  const char* out;
  const char* ovf;
  char* done;
};
__device__ __forceinline__ const char* run_data(const Ctx& c, u32 rows, int i) {
  const u32 g = (rows >> (8 * i)) & 255u; return c.ws + x_panel(c, g >> 5, (g >> 3) & 3) + (g & 7u) * 4096;
}
__device__ __forceinline__ const char* flag_row(const Ctx& c, const Task& t, int nr, int ng) {
  const int i = (c.lane >> 3) & 3;
  const u32 g = (t.rows >> (8 * (i & (nr - 1)))) & 255u;
  return c.ws + FLG_OFF + (c.lane < 32 ? (g >> 5) * FLG_B + ((g & 31u) >> 1) * 512 : t.wstream * FLG_B + (t.cg + (i & (ng - 1))) * 512);
}

__device__ __forceinline__ float gelu(float y) {
  const float t = y * (y * y * 0.044715f + 1.0f) * -2.302208198f;
  return y * __builtin_amdgcn_rcpf(1.0f + __builtin_amdgcn_exp2f(t));
}

template <int OFF> __device__ __forceinline__ void glds(const char* g, u32 voff, u32 ldsdst) {
  asm volatile("s_mov_b32 m0, %2\n\ts_nop 0\n\tglobal_load_lds_dwordx4 %0, %1 offset:%3 sc0 sc1" :: "v"(voff), "s"(g), "s"(ldsdst), "n"(OFF) : "memory");
}

#define REM_BITS " sc0 sc1"
template <int OFF> __device__ __forceinline__ void gldr(const char* g, u32 voff, u32 ldsdst) {
  asm volatile("s_mov_b32 m0, %2\n\ts_nop 0\n\tglobal_load_lds_dwordx4 %0, %1 offset:%3" REM_BITS :: "v"(voff), "s"(g), "s"(ldsdst), "n"(OFF) : "memory");
}
__device__ __forceinline__ void gldr_line(const char* g, u32 voff, u32 ldsdst) {
  asm volatile("s_mov_b32 m0, %2\n\ts_mov_b64 exec, 0xff\n\ts_nop 0\n\tglobal_load_lds_dwordx4 %0, %1" REM_BITS "\n\ts_mov_b64 exec, -1"
               :: "v"(voff), "s"(g), "s"(ldsdst) : "memory");
}
__device__ __forceinline__ u32 all_rows(bool p) { u64 m = __ballot(p); m &= m >> 32; m &= m >> 16; m &= m >> 8; return (u32)m & 0xffu; }
__device__ __forceinline__ int poll_ready(const Ctx& c, const char* fp, int ready, int need) {
  const u64 t0 = __builtin_amdgcn_s_memrealtime();
  for (int n = 1;; ++n) {
    const int g0 = ready >> 2;
    int g = g0 + (c.lane & 7); g = g < NKT / 4 ? g : NKT / 4 - 1;
    u4 f;
    asm volatile("global_load_dwordx4 %0, %1, off sc0 sc1\n\ts_waitcnt vmcnt(0)" : "=&v"(f) : "v"(fp + g * 16) : "memory");
    const u32 d1 = f[0] - c.call, d2 = d1 | (f[1] - c.call), d3 = d2 | (f[2] - c.call), d4 = d3 | (f[3] - c.call);
    const u32 m1 = all_rows((int)d1 >= 0), m2 = all_rows((int)d2 >= 0), m3 = all_rows((int)d3 >= 0), m4 = all_rows((int)d4 >= 0);
    const int i = __builtin_ctz(~m4);
    ready = 4 * (g0 + i) + ((m1 >> i) & 1) + ((m2 >> i) & 1) + ((m3 >> i) & 1);
    if (ready >= need) break;
    if ((n & 3) == 0 && give_up(c, t0)) { ready = NKT; break; }
    __builtin_amdgcn_s_sleep(POLL_SLEEP);
  }
  return ready;
}

__device__ __forceinline__ void put_rows(const Ctx& c, LAS char* scr, const Task& t, const f4 (&v)[4], int rb, int cw, bool poison) {
  if ((rb >> 1) >= t.nruns) return;
  u4 o[4];
#pragma unroll
  for (int i = 0; i < 4; ++i) {
    f4 g = {gelu(v[0][i]), gelu(v[1][i]), gelu(v[2][i]), gelu(v[3][i])};
    if (poison) g = (f4){__builtin_nanf(""), __builtin_nanf(""), __builtin_nanf(""), __builtin_nanf("")};
    o[i] = __builtin_bit_cast(u4, g);
  }
  if (t.ovf) {
    const int pc = rb * 4 + 2 * cw;
    pack32(o[0], o[1], scr, t.out + rb * SL_B + 2 * cw * PF_B, t.ovf + pc * 2048, c.lane);
    pack32(o[2], o[3], scr, t.out + rb * SL_B + (2 * cw + 1) * PF_B, t.ovf + (pc + 1) * 2048, c.lane);
  } else {
    const char* ob = t.out + (size_t)(((t.rows >> (8 * (rb >> 1))) & 255u) * 32 + (rb & 1) * 16) * (NLOC * 4) + cw * 256;
    const u32 oo = (u32)(c.lane >> 4) * (4 * NLOC * 4) + (u32)(c.lane & 15) * 16u;
#pragma unroll
    for (int i = 0; i < 4; ++i) st16_out<0>(ob + (size_t)i * (NLOC * 4), oo, o[i]);
  }
}
#define MF(A, B, m, j) acc[(m) % TM][j] = __builtin_amdgcn_mfma_f32_16x16x32_f16(A[(m) % TM], B[j], acc[(m) % TM][j], 0, 0, 0)

template <int TM, bool Q>
__device__ __forceinline__ void gemm_task(const Ctx& c, LAS char* lds, const Task& t, int sb) {
  constexpr int AB = Q ? 16384 : TM * 4096, ST = AB + 16384;
  constexpr int NL = TM == 4 || Q ? 4 : 3, NB = Q ? NBQ : TM == 4 ? NB4 : NB2;
  constexpr int KS = Q ? 2 : 1, NS = NKT / KS;
  const int wid = c.wid, lane = c.lane;
  const int kh = Q ? (wid >> 1) & 1 : wid >> 2, wm = Q ? wid & 1 : (wid >> 1) & 1, wn = Q ? wid >> 2 : wid & 1, fr = lane & 15, fq = lane >> 4;
  const u32 lds0 = (u32)(uintptr_t)lds, voff = (u32)lane * 16u;
  const char* gb = c.ws + DATA_OFF + (size_t)(4 * t.wstream + (t.cg >> 2)) * PANEL_B + (t.cg & 3) * 8192;
  const char* g0 = Q ? (wid < 4 ? run_data(c, t.rows, wid & 1) : gb + (wid & 1) * 4096) + ((wid >> 1) & 1) * KT_B
                 : TM == 4 ? (wid < 4 ? run_data(c, t.rows, wid) : gb + (wid & 3) * 4096) : gb + wid * 2048;
  const char* g1 = run_data(c, t.rows, wid >> 2) + (wid & 3) * 1024;
  const u32 s0 = NL == 4 ? lds0 + wid * 4096 : lds0 + AB + wid * 2048, s1 = lds0 + wid * 1024;
  const char* fp = flag_row(c, t, TM, Q ? 1 : 2);
  const u32 pa = lds0 + (Q ? wn * 8192 : 0) + wm * (TM * 2048) + kh * 1024 + (fr * 4 + (fq ^ ((-(fr >> 2)) & 3))) * 16;
  const u32 pb = lds0 + AB + wn * 8192 + kh * 1024 + lane * 16;
  f4 acc[TM][4];
#pragma unroll
  for (int m = 0; m < TM; ++m)
#pragma unroll
    for (int j = 0; j < 4; ++j) acc[m][j] = (f4){0.f, 0.f, 0.f, 0.f};
  h8 A0[TM], B0[4], A1[TM], B1[4];
  int ready = 0;
#define GL(i, kt, off) do { if (!more) break; if (NL == 4) glds<(i) * 1024>(g0 + (size_t)(kt) * (KS * KT_B), voff, s0 + (off)); \
    else if ((i) < 2) glds<((i) & 1) * 1024>(g0 + (size_t)(kt) * KT_B, voff, s0 + (off)); \
    else if ((i) == 2) glds<0>(g1 + (size_t)(kt) * KT_B, voff, s1 + (off)); } while (0)
#define RA(A, m, off) A[(m) % TM] = *(const LAS h8*)(uintptr_t)(pa + (off) + ((m) % TM) * 2048)
#define RB(B, j, off) B[j] = *(const LAS h8*)(uintptr_t)(pb + (off) + (j) * 2048)
  if (wid == 0) ready = poll_ready(c, fp, 0, KS * NB);
  raw_barrier(); if (wid == 0) STAMP(sb);
  { constexpr bool more = true;
#pragma unroll
    for (int i = 0; i < NB; ++i) { GL(0, i, i * ST); GL(1, i, i * ST); GL(2, i, i * ST); GL(3, i, i * ST); } }
  wait_vm<(NB - 1) * NL>(); raw_barrier();
#pragma unroll
  for (int m = 0; m < TM; ++m) RA(A0, m, 0);
#pragma unroll
  for (int j = 0; j < 4; ++j) RB(B0, j, 0);
  u32 cur = 0;
#define STEP(s, AC, BC, AN, BN, VM) do { \
    const u32 nb = cur + ST == NB * ST ? 0 : cur + ST; \
    const int nx = (s) + NB; \
    if (more && wid == 0 && KS * (nx + 1) > ready) { ready = poll_ready(c, fp, ready, KS * (nx + 1)); STAMP(sb + 1); DBG_PUT(sb + 3, KS * (s)); } \
    FENCE(); __builtin_amdgcn_s_waitcnt(waitcnt(VM, 0)); raw_barrier(); SB(); \
    if (TM == 4) { \
      MF(AC, BC, 0, 0); RA(AN, 0, nb); SB(); MF(AC, BC, 0, 1); RB(BN, 0, nb); SB(); MF(AC, BC, 0, 2); RA(AN, 1, nb); SB(); MF(AC, BC, 0, 3); RB(BN, 1, nb); SB(); \
      MF(AC, BC, 1, 0); RA(AN, 2, nb); SB(); MF(AC, BC, 1, 1); RB(BN, 2, nb); SB(); MF(AC, BC, 1, 2); RA(AN, 3, nb); SB(); MF(AC, BC, 1, 3); RB(BN, 3, nb); FENCE(); SB(); \
      MF(AC, BC, 2, 0); GL(0, nx, cur); SB(); MF(AC, BC, 2, 1); MF(AC, BC, 2, 2); GL(1, nx, cur); SB(); MF(AC, BC, 2, 3); MF(AC, BC, 3, 0); GL(2, nx, cur); SB(); \
      MF(AC, BC, 3, 1); MF(AC, BC, 3, 2); GL(3, nx, cur); SB(); MF(AC, BC, 3, 3); SB(); \
    } else { \
      MF(AC, BC, 0, 0); RA(AN, 0, nb); RB(BN, 0, nb); SB(); MF(AC, BC, 0, 1); RB(BN, 1, nb); RA(AN, 1, nb); SB(); MF(AC, BC, 0, 2); RB(BN, 2, nb); RB(BN, 3, nb); FENCE(); SB(); \
      MF(AC, BC, 0, 3); GL(0, nx, cur); SB(); MF(AC, BC, 1, 0); GL(1, nx, cur); SB(); MF(AC, BC, 1, 1); GL(2, nx, cur); SB(); MF(AC, BC, 1, 2); GL(3, nx, cur); SB(); MF(AC, BC, 1, 3); SB(); \
    } \
    cur = nb; } while (0)
  for (int s = 0; s < ((NS - NB) & ~1); s += 2) {
    constexpr bool more = true;
    STEP(s, A0, B0, A1, B1, (NB - 2) * NL);
    STEP(s + 1, A1, B1, A0, B0, (NB - 2) * NL);
  }
  if (NB & 1) { constexpr bool more = true; STEP(NS - NB - 1, A0, B0, A1, B1, (NB - 2) * NL); }
  u32 bad;
  { constexpr bool more = false; const u32 z = 0;
    asm volatile("global_load_dword %0, %1, %2 sc1" : "=v"(bad) : "v"(z), "s"(c.ws + TMO_OFF) : "memory");
    if (NB >= 6) STEP(NS - 6, A0, B0, A1, B1, 4 * NL + 1);
    if (NB >= 5) STEP(NS - 5, A1, B1, A0, B0, 3 * NL + 1);
    if (NB >= 4) STEP(NS - 4, A0, B0, A1, B1, 2 * NL + 1);
    STEP(NS - 3, A1, B1, A0, B0, NL + 1);
    STEP(NS - 2, A0, B0, A1, B1, 0);
    asm volatile("" : "+v"(bad));
    STEP(NS - 1, A1, B1, A0, B0, 0);
  }
#undef STEP
#undef GL
#undef RA
#undef RB
  if (wid == 0) STAMP(sb + 2);
  constexpr int H = TM / 2;
  LAS f4* ex = (LAS f4*)lds + wid * 512 + lane;
  if (Q) {
    typedef float f2 __attribute__((ext_vector_type(2)));
#pragma unroll
    for (int i = 0; i < 8; ++i) ex[i * 64] = acc[(i >> 2) % TM][i & 3];
    wait_lgkm(); raw_barrier();
    f4 g0_, g1_;
#pragma unroll
    for (int j = 0; j < 4; ++j) {
      f2 s = {0.f, 0.f};
#pragma unroll
      for (int k = 0; k < 4; ++k) s += *(const LAS f2*)(lds + (2 * k + wm) * 8192 + (wn * 4 + j) * 1024 + lane * 16 + kh * 8);
      g0_[j] = bad == c.call ? __builtin_nanf("") : gelu(s[0]); g1_[j] = bad == c.call ? __builtin_nanf("") : gelu(s[1]);
    }
    const int rb = 2 * wm + wn, pc = 2 * (t.cg & 1) + kh;
    if (wm < t.nruns)
      pack32(__builtin_bit_cast(u4, g0_), __builtin_bit_cast(u4, g1_), lds + 65536 + wid * 2048, t.out + rb * SL_B + pc * PF_B, t.ovf + (rb * 4 + pc) * 2048, lane);
    wait_vm<0>(); wait_lgkm(); raw_barrier();
    if (threadIdx.x == 0) st_sys((GAS u32*)t.done, c.call);
    return;
  }
  LAS f4* ey = (LAS f4*)lds + (wid ^ 4) * 512 + lane;
  f4 v[H][4];
  if (kh == 0) {
#pragma unroll
    for (int m = 0; m < H; ++m)
#pragma unroll
      for (int j = 0; j < 4; ++j) { ex[(m * 4 + j) * 64] = acc[H + m][j]; v[m][j] = acc[m][j]; }
  } else {
#pragma unroll
    for (int m = 0; m < H; ++m)
#pragma unroll
      for (int j = 0; j < 4; ++j) { ex[(m * 4 + j) * 64] = acc[m][j]; v[m][j] = acc[H + m][j]; }
  }
  wait_lgkm(); raw_barrier();
#pragma unroll
  for (int m = 0; m < H; ++m) {
#pragma unroll
    for (int j = 0; j < 4; ++j) v[m][j] += ey[(m * 4 + j) * 64];
    put_rows(c, lds + 65536 + wid * 2048, t, v[m], wm * TM + kh * H + m, wn, bad == c.call);
  }
  if (t.done) wait_vm<0>();
  wait_lgkm(); raw_barrier();
  if (t.done && threadIdx.x == 0) st_sys((GAS u32*)t.done, c.call);
}
#undef MF

__global__ __launch_bounds__(512, 2) void ag_gemm(Params p) {
  extern __shared__ __attribute__((aligned(1024))) char smem[];
  LAS char* lds = (LAS char*)smem;
  const int rank = p.rank & 7, me = rank & 3, og = (rank & 4) ^ 4;
  Ctx c;
  c.ws = p.ws[rank];
  c.wid = __builtin_amdgcn_readfirstlane(threadIdx.x >> 6); c.lane = threadIdx.x & 63;
  STAMP(0);
  const int xcd = blockIdx.x & 7, slot = blockIdx.x >> 3;
  c.tmo = (GAS u32*)(c.ws + TMO_OFF);
  auto get_call = [&]() {
    GAS u32* ctr = (GAS u32*)(c.ws + CTR_OFF + (size_t)blockIdx.x * 128);
    u32 call = (u32)__builtin_amdgcn_readfirstlane(ld_agent(ctr)) + 1u;
    asm volatile("" : "+s"(call) :: "memory");
    raw_barrier();
    if (threadIdx.x == 0) st_agent(ctr, call);
    c.call = call;
    c.par = PUB_OFF + (size_t)(call & 1u) * PAR_B;
  };

  if (slot < NCS) {
    const u32 ld = (u32)c.lane * 16u;
    const u32 tab = (u32)(uintptr_t)lds + 7 * 16384;
    if (threadIdx.x < 336) {
      const u32 i = threadIdx.x;
      u32 v = (i & 3u) | ((i & 12u) << 6) | ((i & 48u) << 12) | ((i & 192u) << 18);
      if (i >= 256) { v = 0; for (u32 k = 0, n = 0; k < 4; ++k) v |= (((i >> k) & 1u) ? n++ : 12u) << (8 * k); }
      if (i >= 272) { const u32 lo = (i - 272u) & 31u;
        v = ((((lo - 1u) ^ (lo + 3u)) & 31u) << 2) | ((((lo - 2u) ^ (lo + 3u)) & 31u) << 10) | (i >= 304 ? (((lo - 3u) ^ (lo + 3u)) & 31u) << 18 : 0u) | ((lo + 3u) << 26) | 0x01000000u; }
      *(LAS u32*)(uintptr_t)(tab + (i < 256 ? 4u * i : i < 272 ? 1024u + 16u * (i - 256u) : 192u + 4u * i)) = v;
      wait_lgkm();
    }
    if (blockIdx.x < NP) {
      const int u = blockIdx.x, d = c.wid & 3;
      if (c.wid < 4) {
        f4 rw[16]; u4 cv[8];
        w_load(rw, w_src(p.w, 4 * d + (u & 3), u >> 2), w_lane_off(c.lane));
        get_call();
        wait_vm<0>(); pin4(rw); pin4(rw + 4); pin4(rw + 8); pin4(rw + 12);
        w_cvt(cv, rw);
        const char* e = p.ws[og + d] + DATA_OFF + (size_t)(36 + me) * PANEL_B + (size_t)(u >> 2) * KT_B + (u & 3) * 8192;
#pragma unroll
        for (int i = 0; i < 8; ++i) { st16_wt<0>(e + i * 1024, ld, cv[i]); wait_vm<PWW>(); }
      } else {
        f4 rx[4];
        x_load(rx, x_src(p.x, 4 * (u & 3) + d, u >> 2), x_lane_off(c.lane));
        get_call();
        wait_vm<0>(); pin4(rx);
        const u4 x0 = cvt8(rx[0], rx[1]), x1 = cvt8(rx[2], rx[3]);
        const size_t xo = x_panel(c, rank, 0) + (size_t)(u >> 2) * KT_B + (u & 3) * 8192 + d * 2048;
#pragma unroll
        for (int i = 0; i < 6; ++i) { st16_wt<0>(p.ws[rank ^ (i % 3 + 1)] + xo + (i / 3) * 1024, ld, i < 3 ? x0 : x1); wait_vm<PWX>(); }
      }
    } else get_call();
    STAMP(6);
    auto where = [&](int l, int u) {
      const int d = l - 3, peer = (l < 3 ? rank ^ (l + 1) : og + d) & 7;
      const Unit un = l < 3 ? in_unit(u) : out_unit(u);
      u4 r = {(u32)un.kind, 0u, 0u, 0u};
      if (un.kind != 3) {
        const int w = un.kind == 2, odd = un.kind == 4 && l >= 3;
        r[1] = (u32)((w * NKT + un.kt + odd) * 16 + (w ? 4 * me : 0) + un.grp);
        r[2] = (u32)((w ? DATA_OFF + (size_t)(36 + d) * PANEL_B : x_panel(c, peer, un.grp >> 2)) + (size_t)un.kt * KT_B + (un.grp & 3) * 8192 + odd * 4096);
        r[3] = (u32)(FLG_OFF + (w ? 9 : peer) * FLG_B + (((w ? 4 * d : 0) + un.grp) * NKT + un.kt) * 4) + (un.kind == 4);
      } else {
        const int sl = out_slot(u), tl = slot_tile(sl), rb = sl & (sl < 400 ? 7 : 3);
        r[1] = (u32)(me * NSS + sl);
        r[2] = ((((tile_rows(tl, peer, me) >> (8 * (rb >> 1))) & 255u) * 32 + (rb & 1) * 16) * NLOC + 256 * d + 128 * (tl & 1)) * 4;
      }
      return r;
    };
    LAS u4* post = (LAS u4*)(lds + LDS_B - 256);
    if (c.wid == 7) {
      const GAS u32* cvp = (const GAS u32*)(c.ws + CVT_OFF + (rank ^ ((c.lane < 56 ? c.lane >> 3 : 6) + 1)) * 256);
      int inn = blockIdx.x + NP + NC, nxt = inn;
      bool busy = true;
      int cvt = 0, ready = 0, dead = 0;
      u64 beyond = 0;
      int ein = -1, eout = -1;
      auto look = [&]() {
        if (inn < 256 || nxt < 256) {
          const int u = (c.lane < 3 ? inn : nxt) & 255, pr = (c.lane < 3 ? rank ^ (c.lane + 1) : og + ((c.lane - 3) & 3)) & 7;
          const u32 m = (u32)__ballot((int)(ld_sys((const GAS u32*)(c.ws + EFL_OFF) + pr * 64 + (u >> 3) * 2 + ((u >> 1) & 1)) - c.call) >= 0);
          if (inn < 256 && (m & 7u) == 7u) ein = inn;
          if (nxt < 256 && (m & 0x78u) == 0x78u) eout = nxt;
        }
        if (cvt < 64) {
          const int g = cvt + (c.lane & 7);
          cvt += __builtin_ctz(~all_rows((int)(ld_sys(cvp + (g < 64 ? g : 63)) - c.call) >= 0));
        }
        if (nxt >= UX && ready < 62) {
          const GAS u32* stp = (const GAS u32*)(c.ws + OST_OFF) + min(ready + c.lane, 61) * 4;
          const u64 m = __ballot((int)((ld_sys(stp) - c.call) | (ld_sys(stp + 1) - c.call) | (ld_sys(stp + 2) - c.call) | (ld_sys(stp + 3) - c.call)) >= 0);
          const int n = m == ~0ull ? 63 : __builtin_ctzll(~m);
          ready += n; beyond = m >> n;
        }
      };
      auto staged = [&](int t) { return t < ready || ((beyond >> (t - ready)) & 1) != 0; };
#define AVAIL_IN(u) ((u) < NP + 2 * NC || (u) == ein || ((u) < UI && (in_unit(u).kt >> 2) + ((u) < 512 ? 0 : 32) < cvt))
#define AVAIL_OUT(u) ((u) < NP + 2 * NC || (u) == eout || ((u) < UX ? (out_unit(u).kt >> 2) + ((u) < UW ? 0 : 32) < cvt : (u) < UO && staged(slot_tile(out_slot(u)))))
      wait_vm<0>();
      for (int it = 1;; ++it) {
        if ((inn < UI && !AVAIL_IN(inn)) || (nxt < UO && !AVAIL_OUT(nxt))) look();
        if (!busy && (inn < UI || nxt < UO)) {
          const u64 t0 = __builtin_amdgcn_s_memrealtime();
          for (int n = 1; !AVAIL_IN(inn) && !AVAIL_OUT(nxt); ++n) {
            if ((n & 15) == 0 && give_up(c, t0)) { cvt = 64; ready = 62; dead = 2; break; }
            look();
          }
        }
        const bool take_in = AVAIL_IN(inn), take_out = AVAIL_OUT(nxt);
        const int in0 = take_in ? inn : UI, out0 = take_out ? nxt : UO;
        const bool stop = inn == UI && nxt == UO;
        if (c.lane < 7) {
          u4 r = where(c.lane, c.lane < 3 ? in0 : out0);
          r[0] = ((c.lane < 3 ? take_in : take_out) ? r[0] : 0u) | (u32)((stop ? 1 : 0) | dead) << 4;
          post[(it & 1) * 8 + c.lane] = r;
        }
        wait_lgkm(); raw_barrier();
        if (blockIdx.x < NP && it == 1 && c.lane < 7) {
          const int u = blockIdx.x;
          const char* pw = p.ws[(c.lane < 3 ? rank ^ (c.lane + 1) : og + c.lane - 3) & 7];
          st_sys((GAS u32*)(pw + FLG_OFF + (c.lane < 3 ? rank * FLG_B + ((u & 3) * NKT + (u >> 2)) * 4
                                                       : 9 * FLG_B + ((4 * me + (u & 3)) * NKT + (u >> 2)) * 4)), c.call);
        }
        if (it >= 12) STAMP(8 + (it < 51 ? it - 12 : 39));
        if (stop) break;
        busy = take_in || take_out;
        u32 tk = 0;
        if (c.lane < 2 && (c.lane == 0 ? take_in : take_out))
          tk = __hip_atomic_fetch_add((GAS u32*)(c.ws + CLM_OFF + c.lane * 256), 1u, __ATOMIC_RELAXED, __HIP_MEMORY_SCOPE_AGENT);
        if (take_in) {
          const u32 v = __builtin_amdgcn_readlane(tk, 0) - (c.call - 1u) * (u32)(UI - NP - NC) + NP + 2 * NC;
          inn = v < UI ? (int)v : UI;
        }
        if (take_out) {
          const u32 v = __builtin_amdgcn_readlane(tk, 1) - (c.call - 1u) * (u32)(UO - NP - NC) + NP + 2 * NC;
          nxt = v < UO ? (int)v : UO;
        }
      }
      STAMP(4);
      return;
    }
    const bool ing = c.wid < 3;
    const int d = c.wid - 3, peer = (ing ? rank ^ (c.wid + 1) : og + d) & 7;
    const char* pw = p.ws[peer] + c.par;
    LAS char* img = lds + c.wid * 16384;
    const u32 lo = (u32)(c.lane >> 4) * (4 * NLOC * 4) + (u32)(c.lane & 15) * 16u;
    const u32 lt = c.lane < 56 ? ld : 880u;
    auto uniform = [&](u4 r) { return (u4){(u32)__builtin_amdgcn_readfirstlane(r[0]), (u32)__builtin_amdgcn_readfirstlane(r[1]),
                                           (u32)__builtin_amdgcn_readfirstlane(r[2]), (u32)__builtin_amdgcn_readfirstlane(r[3])}; };
#define SRC(r) (pw + ((r)[0] == 3 ? STG_OFF : PK_OFF) + (size_t)(r)[1] * SL_B + 128)
#define OVF(r) (pw + ((r)[0] == 3 ? SOV_OFF : OVF_OFF) + (size_t)(r)[1] * 8192)
#define DST(r) (((r)[0] == 3 ? (const char*)p.out[rank] : c.ws) + (r)[2])
    const char* csrc = nullptr; const char* cdst = nullptr; const char* covf = nullptr;
    u32 cflag = 0, pflag = 0;
    int top = 0; u32 hdr = 0;
    int have = 0, flag = 0;
    for (int it = 0;; ++it) {
      LAS char* ci = img + ((it + 1) & 1) * 8192;
      const u32 cia = (u32)(uintptr_t)ci + 128u, nia = (u32)(uintptr_t)img + (u32)(it & 1) * 8192u + 128u;
#define ASK(s, i, k) do { gldr<((k) & 3) * 1024>((s) + ((k) >> 2) * 4096, ld, (i) + ((k) >> 2) * 4096); \
        gldr<(((k) + 1) & 3) * 1024>((s) + ((k) >> 2) * 4096, ld, (i) + ((k) >> 2) * 4096); } while (0)
#define ASK_LAST(s, i) gldr<2048>((s) + 4096, lt, (i) + 4096)
#define A1(s, i, k) gldr<((k) & 3) * 1024>((s) + ((k) >> 2) * 4096, ld, (i) + ((k) >> 2) * 4096)
#define A2(s, i, k) [&]() { gldr<(((k) + 1) & 3) * 1024>((s) + ((k) >> 2) * 4096, ld, (i) + ((k) >> 2) * 4096); }
#define PUT(k, MID) do { u4 a_, b_; \
        if (have == 3) { unpack32(a_, b_, ci + 128 + (k) * PF_B, tab, covf + (k) * 2048, c.lane, MID); \
          if (flag & 2) a_ = b_ = (u4){0x7fc00000u, 0x7fc00000u, 0x7fc00000u, 0x7fc00000u}; \
          st16_out<((k) >> 1) * 256>(cdst + ((k) & 1) * (2 * NLOC * 4), lo, a_); st16_out<((k) >> 1) * 256>(cdst + (((k) & 1) * 2 + 1) * (NLOC * 4), lo, b_); \
        } else { if ((k) == 0) { hdr = __builtin_amdgcn_readfirstlane(*(const LAS u32*)(ci + 508)); top = 508; } \
          unpack16(a_, b_, k, ci, tab, top, hdr, covf + (k) * 2048, c.lane, MID); \
          const char* e_ = cdst + (have == 2 ? ((k) & 1) * 4096 + ((k) >> 1) * 1024 : ((k) & 1) * 2048 + ((k) >> 1) * (have == 4 ? (int)KT_B : 4096)); \
          st16_wt<0>(e_, ld, a_); st16_wt<0>(e_ + (have == 2 ? 2048 : 1024), ld, b_); } } while (0)
#define PUBLISH() do { if (pflag && c.lane <= (int)(pflag & 1u)) st_sys((GAS u32*)(c.ws + (pflag & ~1u)) + c.lane, c.call); } while (0)
      u4 nx;
      if (it == 0) nx = uniform(where(c.wid, blockIdx.x + NP));
      else {
        if (have) { ASK(csrc, cia, 2); wait_vm<2>(); PUBLISH(); PUT(0, []() {}); wait_vm<2>(); }
        raw_barrier();
        nx = uniform(post[(it & 1) * 8 + c.wid]);
        flag = nx[0] >> 4; nx[0] &= 15u;
      }
      if (have) { A1(csrc, cia, 4); PUT(1, A2(csrc, cia, 4)); }
      const int take = nx[0];
      const char* nsrc = SRC(nx); const char* ndst = DST(nx); const char* novf = OVF(nx); const u32 nflag = nx[3];
      if (it < 2) {
        if (it == 0 && blockIdx.x >= NP) { u32 w0 = ld_sys((const GAS u32*)nsrc), w1 = ld_sys((const GAS u32*)ndst); asm volatile("" :: "v"(w0), "v"(w1)); }
        const int nu = blockIdx.x + NP + it * NC;
        const GAS u32* ef = (const GAS u32*)(c.ws + EFL_OFF) + peer * 64 + (nu >> 3) * 2 + ((nu >> 1) & 1);
        const u64 t0 = __builtin_amdgcn_s_memrealtime();
        for (int n = 1; (int)(ld_sys(ef) - c.call) < 0; ++n) if ((n & 15) == 0 && give_up(c, t0)) break;
        STAMP(48 + it * 8 + c.wid);
      }
      if (have) {
        wait_vm<2>(); ASK_LAST(csrc, cia);
        if (have != 3 && (hdr & 64u)) gldr_line(csrc - 128, ld, cia - 128u);
        PUT(2, []() {}); wait_vm<2>();
        pflag = cflag;
        if (take) { A1(nsrc, nia, 0); PUT(3, A2(nsrc, nia, 0)); wait_vm<2>(); }
        else { PUT(3, []() {}); wait_vm<0>(); PUBLISH(); }
      } else if (take) { pflag = 0; ASK(nsrc, nia, 0); wait_vm<0>(); }
      if (flag & 1) break;
      csrc = nsrc; cdst = ndst; covf = novf; cflag = nflag; have = take;
    }
    if (!ing && blockIdx.x < 36) {
      const int u = UO + blockIdx.x;
      const u4 nx = uniform(where(c.wid, u));
      csrc = SRC(nx); cdst = DST(nx); covf = OVF(nx);
      const GAS u32* tf = (const GAS u32*)(c.ws + OST_OFF) + (slot_tile(out_slot(u)) + 10 * (c.lane & 1)) * 4 + d;
      const u64 t0 = __builtin_amdgcn_s_memrealtime();
      for (int n = 1; !__all((int)(ld_sys(tf) - c.call) >= 0); ++n) {
        if ((n & 15) == 0 && give_up(c, t0)) { flag = 2; break; }
        __builtin_amdgcn_s_sleep(LAST_SLEEP);
      }
      STAMP(3);
      LAS char* ci = img;
      const u32 cia = (u32)(uintptr_t)ci + 128u;
      have = 3;
      ASK(csrc, cia, 0); ASK(csrc, cia, 2); wait_vm<2>();
      ASK(csrc, cia, 4); PUT(0, []() {}); wait_vm<4>();
      ASK_LAST(csrc, cia); PUT(1, []() {}); wait_vm<5>();
      PUT(2, []() {}); wait_vm<4>(); PUT(3, []() {});
      STAMP(5);
    }
    return;
  }
  {
    const u32 ld = (u32)c.lane * 16u, lx = x_lane_off(c.lane), lw = w_lane_off(c.lane);
    const u64 peer = (u64)p.ws[(rank ^ ((c.lane & 7) + 1)) & 7];
    LAS u32* lut = (LAS u32*)(lds + 32768);
    LAS unsigned short* sizes = (LAS unsigned short*)(lds + 32768 + 64);
    if (threadIdx.x < 16) {
      u32 sel = 12 * B1;
      for (int k = 3; k >= 0; --k) if ((threadIdx.x >> k) & 1) sel = (sel << 8) | (u32)k;
      lut[threadIdx.x] = sel;
      wait_lgkm();
    }
    int j = blockIdx.x - NC;
    for (int k = 0; k < CVT_DELAY * (j < NP * 5 / 4 ? 1 : (j - NP * 5 / 4) / (NC * 5 / 4)); ++k) __builtin_amdgcn_s_sleep(16);
    auto job = [&](int n) { return n < 640 ? (n / 10) * 16 + (n % 10 < 2 ? n % 10 : n % 10 + 6) : ((n - 640) / 6) * 16 + 2 + (n - 640) % 6; };
    f4 r[8];
#define CL(i) do { const bool w_ = ((i) & 8) != 0; \
      const char* s_ = w_ ? w_src(p.w, 2 * ((i) & 7) + (c.wid & 1), 2 * ((i) >> 4) + ((c.wid >> 1) & 1)) + (size_t)(c.wid >> 2) * 32 * NLOC * 4 \
                          : x_src(p.x, 8 * ((i) & 7) + c.wid, 2 * ((i) >> 4)); \
      const u32 l_ = w_ ? lw : lx, t_ = w_ ? NLOC * 4 : 64; const char* z_ = s_ + (w_ ? 16 * NLOC * 4 : 256); \
      ld16<0>(r[0], s_, l_); ld16<0>(r[1], s_, l_ + t_); ld16<0>(r[2], s_, l_ + 2 * t_); ld16<0>(r[3], s_, l_ + 3 * t_); \
      ld16<0>(r[4], z_, l_); ld16<0>(r[5], z_, l_ + t_); ld16<0>(r[6], z_, l_ + 2 * t_); ld16<0>(r[7], z_, l_ + 3 * t_); } while (0)
    { const int i = job(j); CL(i); }
    get_call();
    wait_vm<0>(); pin4(r); pin4(r + 4);
    for (; j < 1024; j += NLOCAL) {
      const int i = job(j);
      const int half = i & 7, w = (i >> 3) & 1, grp = (i >> 5) + (j < 640 ? 0 : 32), kt0 = 2 * (i >> 4);
      if (c.wid < 4) *(LAS u2*)(lds + c.wid * 8192 + c.lane * 8) = (u2){0u, 0u};
      if (j < 2 * NLOCAL) STAMP(40 + 4 * (j >= NLOCAL));
      u4 cv[4];
      if (w) {
#pragma unroll
        for (int q = 0; q < 4; ++q) cv[q] = cvt8(r[0][q], r[1][q], r[2][q], r[3][q], r[4][q], r[5][q], r[6][q], r[7][q]);
      } else {
#pragma unroll
        for (int q = 0; q < 4; ++q) cv[q] = cvt8(r[2 * q], r[2 * q + 1]);
      }
#pragma unroll
      for (int q = 0; q < 4; ++q) asm volatile("" : "+v"(cv[q]));
      { const int n = job(j + NLOCAL < 1024 ? j + NLOCAL : j); CL(n); }
      const int blk = w ? 4 * (2 * half + (c.wid & 1)) : 8 * half + c.wid;
      const char* dd = c.ws + (w ? DATA_OFF + (size_t)(32 + (blk >> 4)) * PANEL_B : x_panel(c, rank, blk >> 4))
                     + (size_t)(kt0 + (w ? (c.wid >> 1) & 1 : 0)) * KT_B + (blk & 15) * 2048 + (w ? c.wid >> 2 : 0) * 1024;
      st16_wt<0>(dd, ld, cv[0]); st16_wt<0>(dd + (w ? 2048 : 1024), ld, cv[1]);
      st16_wt<0>(dd + (w ? 4096 : KT_B), ld, cv[2]); st16_wt<0>(dd + (w ? 6144 : KT_B + 1024), ld, cv[3]);
      const bool two = !w && (blk >> 2) == 11;
      const int u0 = w ? c.wid & 3 : two ? (c.wid & 2) + 1 : c.wid >> 2, u1 = (w || two) ? u0 : u0 + 2;
      const int k0 = w ? 2 * (c.wid >> 2) : two ? c.wid & 1 : c.wid & 3, k1 = w ? k0 + 1 : two ? k0 + 2 : k0;
      const Coded a = code16(cv[0], cv[1], lut), b = code16(cv[2], cv[3], lut);
      if (c.lane == 0) { sizes[4 * u0 + k0] = a.n2 < 0 ? 0xffff : a.n2 + a.n3; sizes[4 * u1 + k1] = b.n2 < 0 ? 0xffff : b.n2 + b.n3; }
      wait_lgkm(); raw_barrier();
      auto size_of = [&](int u) { const u2 v = *(const LAS u2*)(sizes + 4 * u); return (u64)(u32)__builtin_amdgcn_readfirstlane(v[0]) | (u64)(u32)__builtin_amdgcn_readfirstlane(v[1]) << 32; };
      const size_t un = (size_t)((w * NKT + kt0) * 16 + 2 * half);
      const char* ov = c.ws + c.par + OVF_OFF + un * 8192;
      place16(a, cv[0], cv[1], k0, size_of(u0), lds + u0 * 8192, ov + __builtin_amdgcn_readfirstlane(((u0 >> 1) * 16 + (u0 & 1)) * 8192 + k0 * 2048), c.lane);
      place16(b, cv[2], cv[3], k1, size_of(u1), lds + u1 * 8192, ov + __builtin_amdgcn_readfirstlane(((u1 >> 1) * 16 + (u1 & 1)) * 8192 + k1 * 2048), c.lane);
      wait_lgkm(); raw_barrier();
      {
        const int u = c.wid >> 1, h = 4 * (c.wid & 1);
        const char* pk = c.ws + c.par + PK_OFF + (un + (u >> 1) * 16 + (u & 1)) * SL_B + h * 1024;
        const LAS char* im = lds + u * 8192 + h * 1024 + ld;
        st16_wt<0>(pk, ld, *(const LAS u4*)im); st16_wt<1024>(pk, ld, *(const LAS u4*)(im + 1024)); st16_wt<2048>(pk, ld, *(const LAS u4*)(im + 2048));
        if (h == 0) st16_wt<3072>(pk, ld, *(const LAS u4*)(im + 3072));
      }
      wait_vm<0>(); pin4(r); pin4(r + 4); raw_barrier();
      if (j < 2 * NLOCAL) STAMP(41 + 4 * (j >= NLOCAL));
      if (c.wid == 7) {
        if (i < 512 && c.lane < 7 && (w ? (rank ^ (c.lane + 1)) == og + (half >> 1) : half < 2 && c.lane < 3))
          st_sys((GAS u32*)(peer + EFL_OFF + (rank * 64 + kt0 + (half & 1)) * 4), c.call);
        if (c.lane < 4) st_sys((GAS u32*)(c.ws + FLG_OFF + (w ? 8 : rank) * FLG_B) + (2 * half + (c.lane >> 1)) * NKT + kt0 + (c.lane & 1), c.call);
        u32 n = 0;
        if (c.lane == 0) n = __hip_atomic_fetch_add((GAS u32*)(c.ws + CVC_OFF + grp * 128), 1u, __ATOMIC_RELAXED, __HIP_MEMORY_SCOPE_AGENT);
        if ((u32)__builtin_amdgcn_readfirstlane(n) == (j < 640 ? 20u : 12u) * c.call - 1u && c.lane < 7) st_sys((GAS u32*)(peer + CVT_OFF + (rank * 64 + grp) * 4), c.call);
      }
    }
  }
  STAMP(1);
  const int li = slot - NCS;
  if (li >= NGS) return;
  const int hp = xcd & 1, q = xcd >> 1;
  auto local = [&](u32 rows, int nruns, int c1) {
    Task t; t.rows = rows; t.nruns = nruns; t.wstream = 8; t.cg = 4 * q + 2 * c1;
    t.out = (const char*)(p.out[rank] + 64 * t.cg); t.ovf = nullptr; t.done = nullptr;
    return t;
  };
  auto staged = [&](int tl) {
    Task t; t.rows = tile_rows(tl, rank, q); t.nruns = tl >= 70 ? 1 : ((tl >> 1) == 11 || (tl >> 1) == 24) ? 3 : 4; t.wstream = 9; t.cg = 4 * q + 2 * (tl & 1);
    const size_t i = q * NSS + (tl < 50 ? 8 * tl : 400 + 4 * (tl - 50));
    t.out = c.ws + c.par + STG_OFF + i * SL_B + 128; t.ovf = c.ws + c.par + SOV_OFF + i * 8192;
    t.done = p.ws[og + q] + OST_OFF + (tl * 4 + me) * 4;
    return t;
  };
  constexpr int NT = 26 + 27 + 27;
  LAS int* next = (LAS int*)(lds + LDS_B - 64);
  for (int k = li, n = 0; k < NT; ++n) {
    const int c1 = k & 1;
    Task t;
    if (k < 12) t = staged(4 * (k >> 1) + 2 * hp + c1);
    else if (k < 18) t = local(rows128(rank ^ ((k - 10) >> 1), 4 * hp), 4, c1);
    else if (k < 26) t = local(rows128(rank, 8 * ((k - 18) >> 1) + 4 * hp), 4, c1);
    else if (k < 38) t = staged(4 * ((k - 14) >> 1) + 2 * hp + c1);
    else if (k == 38) t = staged(48 + hp);
    else if (k < 45) t = local(rows128(rank ^ ((k - 37) >> 1), 8 + 4 * hp), 4, c1);
    else if (k < 53) t = local(rows128(og + ((k - 45) >> 1), 24 + 4 * hp), 4, c1);
    else if (k < 55) t = local((rows128(rank ^ 1, 16 + 2 * hp) & 0xffffu) | rows128(rank ^ 2, 16 + 2 * hp) << 16, 4, c1);
    else if (k < 61) t = staged(50 + 4 * ((k - 55) >> 1) + 2 * hp + 1 - c1);
    else if (k < 63) t = local((u32)((og + 2 * (k - 61)) * 32 + 23) * 0x0101u + 0x2000u, 2, hp);
    else if (k < 65) t = local(rows128(rank ^ 3, 16 + 2 * hp), 2, c1);
    else if (k < 70) t = local(last_rows(k - 65, rank), k == 69 ? 1 : 2, hp);
    else { t = staged(62 + 2 * ((k - 70) >> 1) + hp); t.cg += c1; t.done += c1 * 160; }
    DBG_PUT(32 + n, k);
    if (k < 55) gemm_task<4, false>(c, lds, t, 8 + 8 * (n < 2 ? n : 2));
    else if (k < 70) gemm_task<2, false>(c, lds, t, 8 + 8 * (n < 2 ? n : 2));
    else gemm_task<2, true>(c, lds, t, 8 + 8 * (n < 2 ? n : 2));
    STAMP(2 + (n < 2 ? n : 2));
    if (threadIdx.x == 64)
      *next = (int)(__hip_atomic_fetch_add((GAS u32*)(c.ws + TSK_OFF + xcd * 128), 1u, __ATOMIC_RELAXED, __HIP_MEMORY_SCOPE_AGENT) - (c.call - 1u) * (u32)NT) + NGS;
    wait_lgkm(); raw_barrier();
    k = __builtin_amdgcn_readfirstlane(*next);
  }
}

}

extern "C" void kernel_launch(int rank, int nranks, ncclComm_t comm,
                              hipStream_t stream,
                              void* const* d_in, const int* in_sizes, int n_in,
                              void* d_out, int out_size,
                              const int* in_dtypes, int out_dtype,
                              void* const* d_out_all,
                              void* const* d_ws_all, size_t ws_bytes) {
  if (nranks != NR || n_in != 2 || ws_bytes < WS_NEED || (unsigned)rank >= (unsigned)NR ||
      in_sizes[0] != NLOC * KDIM || in_sizes[1] != KDIM * NLOC || out_size != KDIM * NLOC) {
    fprintf(stderr, "unexpected problem shape\n");
    return;
  }
  static bool attr = false;
  if (!attr) {
    HIP_CHECK(hipFuncSetAttribute((const void*)ag_gemm, hipFuncAttributeMaxDynamicSharedMemorySize, LDS_B));
    attr = true;
  }
  Params p;
  p.x = (const float*)d_in[0]; p.w = (const float*)d_in[1];
  for (int r = 0; r < NR; ++r) { p.ws[r] = (char*)d_ws_all[r]; p.out[r] = (float*)d_out_all[r]; }
  p.rank = rank; p.pad = 0;
  hipLaunchKernelGGL(ag_gemm, dim3(256), dim3(512), LDS_B, stream, p);
  HIP_CHECK(hipGetLastError());
}
```
